```python
import numpy as np

kernel_name = "gpu_ag_gemm_m2048_k2048_n2048_f32_relu"


ABSMAX_RTOL = 0.001

COMPUTE_DTYPE = "float32"
EPILOGUE = "relu"
IN_QUANT = None
OUT_QUANT = None
IN_DTYPES = {"x": "float32", "w_mat": "float32"}
WORLD_SIZE = 8

def setup_inputs():

    x = np.random.randn(2048, 2048).astype(np.float32)
    w_mat = (np.random.randn(2048, 2048) / np.sqrt(2048)).astype(np.float32)
    return {"x": x, "w_mat": w_mat}

def _epilogue(y):
    return np.maximum(y, 0.0).astype(np.float32)

def reference(x, w_mat):
    y = (x @ w_mat).astype(np.float32)
    return _epilogue(y)
```

```cpp
#include <hip/hip_runtime.h>
#include <rccl/rccl.h>
#include <cstdio>
#include <cstdint>

#define MODE2D 0

#define HIP_CHECK(expr) do { hipError_t _e = (expr); if (_e != hipSuccess) { \
    fprintf(stderr, "HIP error %s at %s:%d\n", hipGetErrorString(_e), __FILE__, __LINE__); \
    return; } } while (0)

namespace {

typedef _Float16 h8 __attribute__((ext_vector_type(8)));
typedef float f4 __attribute__((ext_vector_type(4)));
typedef unsigned u4 __attribute__((ext_vector_type(4)));

constexpr int NR = 8;
constexpr int MM = 2048, KK = 2048;
constexpr int SH = 256;
constexpr int NTHREADS = 512, NBLOCKS = 256;

constexpr unsigned XT_B = 32 * KK * 2;
constexpr unsigned WS_B = 64 * KK * 2;
constexpr unsigned JOB_B = 8192;
constexpr unsigned X_OFF = 0;
constexpr unsigned W_OFF = NR * 8 * XT_B;
constexpr unsigned PAR_B = W_OFF + 8 * WS_B;
constexpr unsigned FL_STR = 128;
constexpr unsigned FX_OFF = 2 * PAR_B;
constexpr unsigned FW_OFF = FX_OFF + 2 * 64 * 16 * FL_STR;
constexpr unsigned FD_OFF = FW_OFF + 2 * 8 * 32 * FL_STR;
constexpr unsigned CT_OFF = FD_OFF + 64 * FL_STR;
constexpr unsigned TM_OFF = CT_OFF + NBLOCKS * 8 * 4;
constexpr unsigned ST_OFF = TM_OFF + 128;
constexpr unsigned WS_END = ST_OFF + NBLOCKS * 8 * 16 * 8;
constexpr unsigned long long DEADLINE = 100000000ull;

struct Params {
  const float* x;
  const float* w;
  char* ws[NR];
  float* out[NR];
  int rank;
  int pad;
};

#define STAMP(i) do {} while (0)

template <int OFF>
__device__ __forceinline__ void ld16(f4& d, const char* a) {
  asm volatile("global_load_dwordx4 %0, %1, off offset:%2" : "=v"(d) : "v"(a), "i"(OFF) : "memory");
}
template <int OFF>
__device__ __forceinline__ void st16_sys(char* a, u4 v) {
  asm volatile("global_store_dwordx4 %0, %1, off offset:%2 sc0 sc1\n\ts_nop 1"
               :: "v"(a), "v"(v), "i"(OFF) : "memory");
}
__device__ __forceinline__ void st_flag(char* a, unsigned v) {
  asm volatile("s_mov_b64 exec, 1\n\tglobal_store_dword %0, %1, off sc0 sc1\n\ts_mov_b64 exec, -1"
               :: "v"(a), "v"(v) : "memory");
}
template <int N>
__device__ __forceinline__ void wait_vm() { asm volatile("s_waitcnt vmcnt(%0)" :: "i"(N) : "memory"); }

struct Job {
  const char* src;
  char* dst;
  char* flag;
};

template <bool ISW>
__device__ __forceinline__ void job_load(f4 (&r)[16], const char* src, int lane) {
  if constexpr (!ISW) {
    const char* a0 = src + (lane & 15) * (KK * 4) + (lane >> 4) * 32;
    const char* a1 = a0 + 16 * (KK * 4);
    ld16<0>(r[0], a0);    ld16<16>(r[1], a0);   ld16<0>(r[2], a1);    ld16<16>(r[3], a1);
    ld16<128>(r[4], a0);  ld16<144>(r[5], a0);  ld16<128>(r[6], a1);  ld16<144>(r[7], a1);
    ld16<256>(r[8], a0);  ld16<272>(r[9], a0);  ld16<256>(r[10], a1); ld16<272>(r[11], a1);
    ld16<384>(r[12], a0); ld16<400>(r[13], a0); ld16<384>(r[14], a1); ld16<400>(r[15], a1);
  } else {
    const char* a0 = src + (lane >> 4) * (8 * SH * 4) + (lane & 15) * 16 + 4096;
    const char* a1 = a0 + 32 * (SH * 4);
    ld16<-4096>(r[0], a0); ld16<-3072>(r[1], a0); ld16<-2048>(r[2], a0); ld16<-1024>(r[3], a0);
    ld16<0>(r[4], a0);     ld16<1024>(r[5], a0);  ld16<2048>(r[6], a0);  ld16<3072>(r[7], a0);
    ld16<-4096>(r[8], a1); ld16<-3072>(r[9], a1); ld16<-2048>(r[10], a1); ld16<-1024>(r[11], a1);
    ld16<0>(r[12], a1);    ld16<1024>(r[13], a1); ld16<2048>(r[14], a1); ld16<3072>(r[15], a1);
  }
}
__device__ __forceinline__ void pin16(f4 (&r)[16]) {
  asm volatile("" : "+v"(r[0]), "+v"(r[1]), "+v"(r[2]), "+v"(r[3]), "+v"(r[4]), "+v"(r[5]), "+v"(r[6]),
               "+v"(r[7]), "+v"(r[8]), "+v"(r[9]), "+v"(r[10]), "+v"(r[11]), "+v"(r[12]), "+v"(r[13]),
               "+v"(r[14]), "+v"(r[15]) :: "memory");
}
template <bool ISW>
__device__ __forceinline__ void job_cvt(u4 (&h)[8], const f4 (&r)[16]) {
#pragma unroll
  for (int f = 0; f < 8; ++f) {
    h8 v;
#pragma unroll
    for (int j = 0; j < 8; ++j) {
      float s;
      if constexpr (!ISW) s = r[2 * f + (j >> 2)][j & 3];
      else s = r[(f >> 2) * 8 + j][f & 3];
      v[j] = (_Float16)s;
    }
    h[f] = __builtin_bit_cast(u4, v);
  }
}
__device__ __forceinline__ void job_store(const u4 (&h)[8], char* dst, int lane) {
  char* a = dst + lane * 16 + 4096;
  st16_sys<-4096>(a, h[0]); st16_sys<-3072>(a, h[1]); st16_sys<-2048>(a, h[2]); st16_sys<-1024>(a, h[3]);
  st16_sys<0>(a, h[4]);     st16_sys<1024>(a, h[5]);  st16_sys<2048>(a, h[6]);  st16_sys<3072>(a, h[7]);
}

template <int NJ, unsigned WMASK, int J, class MK>
__device__ __forceinline__ void job_steps(f4 (&r)[16], u4 (&h)[8], MK mk, int lane, unsigned call) {
  if constexpr (J >= 2) wait_vm<9>(); else wait_vm<8>();
  pin16(r);
  job_cvt<((WMASK >> J) & 1) != 0>(h, r);
  if constexpr (J + 1 < NJ) {
    job_load<((WMASK >> (J + 1)) & 1) != 0>(r, mk(J + 1).src, lane);
    job_store(h, mk(J).dst, lane);
    wait_vm<24>();
    st_flag(mk(J - 1).flag, call);
    job_steps<NJ, WMASK, J + 1>(r, h, mk, lane, call);
  } else {
    job_store(h, mk(J).dst, lane);
    wait_vm<8>();
    st_flag(mk(J - 1).flag, call);
    wait_vm<0>();
    st_flag(mk(J).flag, call);
  }
}
template <int NJ, unsigned WMASK, class MKS, class MK>
__device__ __forceinline__ void run_jobs(MKS mksrc0, MK mkfull, int lane, const unsigned* ctr, unsigned& call) {
  f4 r[16];
  u4 h[8];
  job_load<(WMASK & 1) != 0>(r, mksrc0(), lane);
  unsigned c;
  asm volatile("global_load_dword %0, %1, off sc1\n\ts_waitcnt vmcnt(0)" : "=&v"(c) : "v"(ctr) : "memory");
  call = __builtin_amdgcn_readfirstlane(c) + 1u;
  pin16(r);
  auto mk = [&](int j) { return mkfull(j, call & 1u); };
  job_cvt<(WMASK & 1) != 0>(h, r);
  if constexpr (NJ > 1) {
    job_load<((WMASK >> 1) & 1) != 0>(r, mk(1).src, lane);
    job_store(h, mk(0).dst, lane);
    job_steps<NJ, WMASK, 1>(r, h, mk, lane, call);
  } else {
    job_store(h, mk(0).dst, lane);
    wait_vm<0>();
    st_flag(mk(0).flag, call);
  }
}

__device__ __forceinline__ Job x_job(const Params& p, int dst, int rt, int hc, unsigned par) {
  Job j;
  j.src = (const char*)p.x + (size_t)rt * 32 * (KK * 4) + hc * 128 * 4;
  char* ws = p.ws[dst];
  j.dst = ws + par * PAR_B + X_OFF + (unsigned)(p.rank * 8 + rt) * XT_B + hc * JOB_B;
  j.flag = ws + FX_OFF + ((par * 64 + p.rank * 8 + rt) * 16 + hc) * FL_STR;
  return j;
}
__device__ __forceinline__ Job w_job(const Params& p, int dst, int slice, int slot, int pc, unsigned par) {
  Job j;
  j.src = (const char*)p.w + (size_t)pc * 64 * (SH * 4) + slice * 64 * 4;
  char* ws = p.ws[dst];
  j.dst = ws + par * PAR_B + W_OFF + (unsigned)slot * WS_B + pc * JOB_B;
  j.flag = ws + FW_OFF + ((par * 8 + slot) * 32 + pc) * FL_STR;
  return j;
}

__device__ __forceinline__ int poll_groups(const char* fl, int nfl, int lg, int have, unsigned call,
                                           unsigned* tmo, int lane) {
  const unsigned* a = (lane < nfl) ? (const unsigned*)(fl + lane * FL_STR) : tmo;
  const unsigned long long t0 = __builtin_amdgcn_s_memrealtime();
  for (;;) {
    const unsigned v = __hip_atomic_load(a, __ATOMIC_RELAXED, __HIP_MEMORY_SCOPE_SYSTEM);
    const unsigned long long notset = __ballot(lane < nfl && v < call);
    const int lead = notset ? __builtin_ctzll(notset) : nfl;
    const int n = lead >> lg;
    if (n > have) return n;
    const bool dead = __ballot(lane >= nfl && v == call) != 0ull;
    if (dead || __builtin_amdgcn_s_memrealtime() - t0 > DEADLINE) {
      if (lane == 0) __hip_atomic_store(tmo, call, __ATOMIC_RELAXED, __HIP_MEMORY_SCOPE_AGENT);
      return -1;
    }
    __builtin_amdgcn_s_sleep(1);
  }
}

__global__ void __launch_bounds__(NTHREADS, 2) ag_gemm_kernel(Params p) {
  __shared__ __attribute__((aligned(16))) float lds[8 * 2048 + 64];
  const int lane = threadIdx.x & 63;
  const int wave = __builtin_amdgcn_readfirstlane(threadIdx.x >> 6);
  const int b = blockIdx.x, xcd = b & 7, bi = b >> 3;
  const int rank = p.rank, grp = rank & 4, idx = rank & 3;
  char* const myws = p.ws[rank];
  unsigned* const ctr = (unsigned*)(myws + CT_OFF) + b * 8 + wave;
  unsigned* const tmo = (unsigned*)(myws + TM_OFF);
  STAMP(0);

  if (bi < 16) {
    const int sid = bi * 8 + xcd;
    unsigned call;
    if (wave == 0 && sid < 112) {
      const int link = sid % 7, i = sid / 7;
      const int dst = (rank + 1 + link) & 7;
      run_jobs<8, 0u>([&] { return x_job(p, dst, i & 3, i >> 2, 0).src; },
                      [&](int j, unsigned par) {
                        return x_job(p, dst, (j >> 2) * 4 + (i & 3), (j & 3) * 4 + (i >> 2), par);
                      }, lane, ctr, call);
    } else if (wave == 1) {
      run_jobs<1, 1u>([&] { return w_job(p, rank, sid >> 5, sid >> 5, sid & 31, 0).src; },
                      [&](int, unsigned par) { return w_job(p, rank, sid >> 5, sid >> 5, sid & 31, par); },
                      lane, ctr, call);
    } else if (wave == 2 && (!MODE2D || sid < 96)) {
      run_jobs<1, 0u>([&] { return x_job(p, rank, sid >> 4, sid & 15, 0).src; },
                      [&](int, unsigned par) { return x_job(p, rank, sid >> 4, sid & 15, par); },
                      lane, ctr, call);
    } else {
      return;
    }
    if (lane == 0) __hip_atomic_store(ctr, call, __ATOMIC_RELAXED, __HIP_MEMORY_SCOPE_AGENT);
    STAMP(1);
    return;
  }

  const unsigned call = __builtin_amdgcn_readfirstlane(
                            __hip_atomic_load(ctr, __ATOMIC_RELAXED, __HIP_MEMORY_SCOPE_AGENT)) + 1u;
  if (lane == 0) __hip_atomic_store(ctr, call, __ATOMIC_RELAXED, __HIP_MEMORY_SCOPE_AGENT);
  const unsigned par = call & 1u;
  const int ci = bi - 16;
  const int pair = xcd * 4 + (ci >> 2), slot = ci & 3;
  const int odst = rank, ocol = slot * 64;
  const __amdgpu_buffer_rsrc_t rs = __builtin_amdgcn_make_buffer_rsrc(myws, 0, (int)(2 * PAR_B), 0x00020000);
  typedef __attribute__((address_space(3))) int* lds_int_p;
  const lds_int_p rdy = (lds_int_p)(lds + 8 * 2048);
  int nw = 0;
  const int voff = lane * 16;
  bool bad = false;
  auto wg_wait = [&](const char* fl, int nfl, int lg, int have) {
    if (wave == 0) {
      const int n = poll_groups(fl, nfl, lg, have, call, tmo, lane);
      if (lane == 0) rdy[nw] = n;
    }
    __syncthreads();
    int n = __builtin_amdgcn_readfirstlane(rdy[nw]);
    ++nw;
    if (n < 0) { bad = true; n = 8; }
    return n;
  };

  h8 B[8][4];
  {
    const char* fl = myws + FW_OFF + (par * 8 + slot) * 32 * FL_STR;
    const int sb = par * PAR_B + W_OFF + slot * WS_B + wave * 4096;
#pragma unroll
    for (int c = 0; c < 8; ++c)
#pragma unroll
      for (int e = 0; e < 4; ++e) B[c][e] = h8{};
    int c = 0;
#pragma unroll 1
    while (c < 8) {
      const int n = wg_wait(fl, 32, 2, c);
#pragma unroll
      for (int d = 0; d < 8; ++d)
        if (d >= c && d < n) {
#pragma unroll
          for (int e = 0; e < 4; ++e)
            B[d][e] = __builtin_bit_cast(h8, __builtin_amdgcn_raw_buffer_load_b128(rs, voff + e * 1024, sb + d * 32768, 17));
        }
      c = n;
    }
  }
  STAMP(2);

#pragma unroll 1
  for (int t = 0; t < 2; ++t) {
    const int src = pair >> 2, rt = (pair & 3) + 4 * t;
    const char* fl = myws + FX_OFF + (par * 64 + src * 8 + rt) * 16 * FL_STR;
    const int sb = par * PAR_B + X_OFF + (src * 8 + rt) * XT_B + wave * 2048;
    f4 acc[2][4];
#pragma unroll
    for (int g = 0; g < 2; ++g)
#pragma unroll
      for (int e = 0; e < 4; ++e) acc[g][e] = f4{0.f, 0.f, 0.f, 0.f};
    h8 A[8][2];
#pragma unroll
    for (int d = 0; d < 8; ++d) { A[d][0] = h8{}; A[d][1] = h8{}; }
    int c = 0, done = 0;
    unsigned tm = 0;
#pragma unroll 1
    for (;;) {
#pragma unroll
      for (int d = 0; d < 8; ++d)
        if (d >= done && d < c) {
#pragma unroll
          for (int g = 0; g < 2; ++g)
#pragma unroll
            for (int e = 0; e < 4; ++e)
              acc[g][e] = __builtin_amdgcn_mfma_f32_16x16x32_f16(A[d][g], B[d][e], acc[g][e], 0, 0, 0);
        }
      if (c == 8) break;
      done = c;
      const int n = wg_wait(fl, 16, 1, c);
#pragma unroll
      for (int d = 0; d < 8; ++d)
        if (d >= c && d < n) {
#pragma unroll
          for (int g = 0; g < 2; ++g)
            A[d][g] = __builtin_bit_cast(h8, __builtin_amdgcn_raw_buffer_load_b128(rs, voff + g * 1024, sb + d * 16384, 17));
        }
      c = n;
      tm = __hip_atomic_load(tmo, __ATOMIC_RELAXED, __HIP_MEMORY_SCOPE_AGENT);
    }
    STAMP(3 + 3 * t);
    if (t) __syncthreads();
    {
      float* slab = lds + wave * 2048 + (lane >> 4) * 4 * 64 + (lane & 15) * 4;
#pragma unroll
      for (int g = 0; g < 2; ++g)
#pragma unroll
        for (int r = 0; r < 4; ++r)
          *(f4*)(slab + (g * 16 + r) * 64) = f4{acc[g][0][r], acc[g][1][r], acc[g][2][r], acc[g][3][r]};
    }
    __syncthreads();
    STAMP(4 + 3 * t);
    f4 s = *(const f4*)(lds + threadIdx.x * 4);
#pragma unroll
    for (int v = 1; v < 8; ++v) s += *(const f4*)(lds + v * 2048 + threadIdx.x * 4);
#pragma unroll
    for (int j = 0; j < 4; ++j) s[j] = s[j] < 0.f ? 0.f : s[j];
    if (bad || tm == call) s = f4{__builtin_nanf(""), __builtin_nanf(""), __builtin_nanf(""), __builtin_nanf("")};
    float* o = p.out[odst] + (size_t)(src * SH + rt * 32 + (threadIdx.x >> 4)) * SH + ocol + (threadIdx.x & 15) * 4;
    st16_sys<0>((char*)o, __builtin_bit_cast(u4, s));
    STAMP(5 + 3 * t);
  }
  STAMP(9);
}

}

extern "C" void kernel_launch(int rank, int nranks, ncclComm_t comm, hipStream_t stream,
                              void* const* d_in, const int* in_sizes, int n_in,
                              void* d_out, int out_size,
                              const int* in_dtypes, int out_dtype,
                              void* const* d_out_all,
                              void* const* d_ws_all, size_t ws_bytes) {
  (void)comm; (void)in_dtypes; (void)out_dtype; (void)d_out;
  if (nranks != NR || (unsigned)rank >= (unsigned)NR || n_in != 2 || in_sizes[0] != SH * KK ||
      in_sizes[1] != KK * SH || out_size != MM * SH || ws_bytes < WS_END) {
    fprintf(stderr, "unexpected problem shape\n");
    return;
  }
  Params p;
  p.x = (const float*)d_in[0];
  p.w = (const float*)d_in[1];
  for (int i = 0; i < NR; ++i) { p.ws[i] = (char*)d_ws_all[i]; p.out[i] = (float*)d_out_all[i]; }
  p.rank = rank;
  p.pad = 0;
  hipLaunchKernelGGL(ag_gemm_kernel, dim3(NBLOCKS), dim3(NTHREADS), 0, stream, p);
  HIP_CHECK(hipGetLastError());
}
```

```cpp
#include <hip/hip_runtime.h>
#include <rccl/rccl.h>
#include <cstdio>
#include <cstdint>

#define HIP_CHECK(expr) do { hipError_t _e = (expr); if (_e != hipSuccess) { \
    fprintf(stderr, "HIP error %s at %s:%d\n", hipGetErrorString(_e), __FILE__, __LINE__); \
    return; } } while (0)

#define WPL 8
#define NP 11

namespace {

typedef _Float16 h8 __attribute__((ext_vector_type(8)));
typedef float f4 __attribute__((ext_vector_type(4)));
typedef unsigned u4 __attribute__((ext_vector_type(4)));

constexpr int NR = 8;
constexpr int MM = 2048, KK = 2048;
constexpr int SH = 256;
constexpr int NTHREADS = 512, NBLOCKS = 256;
constexpr int NJ_P = NP * 8;
constexpr int NJ_X = 32 + (16 - NP) * 8;

constexpr unsigned XU_B = 16 * KK * 2;
constexpr unsigned WS_B = 64 * KK * 2;
constexpr unsigned JOB_B = 8192;
constexpr unsigned X_OFF = 0;
constexpr unsigned W_OFF = NR * 16 * XU_B;
constexpr unsigned PAR_B = W_OFF + 8 * WS_B;
constexpr unsigned FL_STR = 128;
constexpr unsigned FX_OFF = 2 * PAR_B;
constexpr unsigned FW_OFF = FX_OFF + 2 * 128 * 8 * FL_STR;
constexpr unsigned FD_OFF = FW_OFF + 2 * 8 * 32 * FL_STR;
constexpr unsigned TK_OFF = FD_OFF + 64 * FL_STR;
constexpr unsigned TM_OFF = TK_OFF + 8 * FL_STR;
constexpr unsigned CT_OFF = TM_OFF + FL_STR;
constexpr unsigned ST_OFF = CT_OFF + NBLOCKS * 8 * 4;
constexpr unsigned WS_END = ST_OFF + NBLOCKS * 8 * 32 * 8;
constexpr unsigned long long DEADLINE = 100000000ull;

struct Params {
  const float* x;
  const float* w;
  char* myws;
  int rank;
  int pad;
  char* ws[NR];
  float* out[NR];
  unsigned tab[16];
};

#define STAMPV(i, v) do {} while (0)
#define STAMP(i) do {} while (0)
#define STAMP_INIT() do {} while (0)
#define STAMP_FLUSH() do {} while (0)

template <int OFF>
__device__ __forceinline__ void ld16(f4& d, const char* a) {
  asm volatile("global_load_dwordx4 %0, %1, off offset:%2" : "=v"(d) : "v"(a), "i"(OFF) : "memory");
}
template <int OFF>
__device__ __forceinline__ void st16_sys(char* a, u4 v) {
  asm volatile("global_store_dwordx4 %0, %1, off offset:%2 sc0 sc1\n\ts_nop 1"
               :: "v"(a), "v"(v), "i"(OFF) : "memory");
}
__device__ __forceinline__ void st_flag(char* a, unsigned v) {
  asm volatile("s_mov_b64 exec, 1\n\tglobal_store_dword %0, %1, off sc0 sc1\n\ts_mov_b64 exec, -1"
               :: "v"(a), "v"(v) : "memory");
}
__device__ __forceinline__ void draw_ticket(unsigned& t, char* a, unsigned one) {
  asm volatile("s_mov_b64 exec, 1\n\tglobal_atomic_add %0, %1, %2, off sc0\n\ts_mov_b64 exec, -1"
               : "=&v"(t) : "v"(a), "v"(one) : "memory");
}
template <int N>
__device__ __forceinline__ void wait_vm() { asm volatile("s_waitcnt vmcnt(%0)" :: "i"(N) : "memory"); }

struct Job {
  const char* src;
  char* dst;
  char* flag;
  bool isw;
};

__device__ __forceinline__ void job_load(f4 (&r)[16], const Job& jb, int lane) {
  if (!jb.isw) {
    const char* a = jb.src + (lane & 15) * (KK * 4) + (lane >> 4) * 32;
    ld16<0>(r[0], a);    ld16<16>(r[1], a);   ld16<128>(r[2], a);  ld16<144>(r[3], a);
    ld16<256>(r[4], a);  ld16<272>(r[5], a);  ld16<384>(r[6], a);  ld16<400>(r[7], a);
    ld16<512>(r[8], a);  ld16<528>(r[9], a);  ld16<640>(r[10], a); ld16<656>(r[11], a);
    ld16<768>(r[12], a); ld16<784>(r[13], a); ld16<896>(r[14], a); ld16<912>(r[15], a);
  } else {
    const char* a0 = jb.src + (lane >> 4) * (8 * SH * 4) + (lane & 15) * 16 + 4096;
    const char* a1 = a0 + 32 * (SH * 4);
    ld16<-4096>(r[0], a0); ld16<-3072>(r[1], a0); ld16<-2048>(r[2], a0); ld16<-1024>(r[3], a0);
    ld16<0>(r[4], a0);     ld16<1024>(r[5], a0);  ld16<2048>(r[6], a0);  ld16<3072>(r[7], a0);
    ld16<-4096>(r[8], a1); ld16<-3072>(r[9], a1); ld16<-2048>(r[10], a1); ld16<-1024>(r[11], a1);
    ld16<0>(r[12], a1);    ld16<1024>(r[13], a1); ld16<2048>(r[14], a1); ld16<3072>(r[15], a1);
  }
}
__device__ __forceinline__ void pin16(f4 (&r)[16]) {
  asm volatile("" : "+v"(r[0]), "+v"(r[1]), "+v"(r[2]), "+v"(r[3]), "+v"(r[4]), "+v"(r[5]), "+v"(r[6]),
               "+v"(r[7]), "+v"(r[8]), "+v"(r[9]), "+v"(r[10]), "+v"(r[11]), "+v"(r[12]), "+v"(r[13]),
               "+v"(r[14]), "+v"(r[15]) :: "memory");
}
__device__ __forceinline__ void job_cvt(u4 (&h)[8], const f4 (&r)[16], bool isw) {
  if (!isw) {
#pragma unroll
    for (int f = 0; f < 8; ++f) {
      h8 v;
#pragma unroll
      for (int j = 0; j < 8; ++j) v[j] = (_Float16)r[2 * f + (j >> 2)][j & 3];
      h[f] = __builtin_bit_cast(u4, v);
    }
  } else {
#pragma unroll
    for (int f = 0; f < 8; ++f) {
      h8 v;
#pragma unroll
      for (int j = 0; j < 8; ++j) v[j] = (_Float16)r[(f >> 2) * 8 + j][f & 3];
      h[f] = __builtin_bit_cast(u4, v);
    }
  }
}
__device__ __forceinline__ void job_store(const u4 (&h)[8], char* dst, int lane) {
  char* a = dst + lane * 16 + 4096;
  st16_sys<-4096>(a, h[0]); st16_sys<-3072>(a, h[1]); st16_sys<-2048>(a, h[2]); st16_sys<-1024>(a, h[3]);
  st16_sys<0>(a, h[4]);     st16_sys<1024>(a, h[5]);  st16_sys<2048>(a, h[6]);  st16_sys<3072>(a, h[7]);
}

template <class MK>
__device__ __forceinline__ void run_jobs(int q0, int q1, int nj, char* tkt, MK mk, int lane, int wave,
                                         unsigned* ctr) {
  f4 r[16];
  u4 h[8];
  job_load(r, mk(q0, 0u), lane);
  unsigned c;
  asm volatile("global_load_dword %0, %1, off sc1\n\ts_waitcnt vmcnt(0)" : "=&v"(c) : "v"(ctr) : "memory");
  const unsigned call = __builtin_amdgcn_readfirstlane(c) + 1u;
  const unsigned par = call & 1u;
  const unsigned tbase = (call - 1u) * (unsigned)(nj - WPL) - 2u * WPL;
  pin16(r);
  STAMP(2);
  char* pflag = (char*)ctr;
  int cur = q0, nxt = q1;
#pragma unroll 1
  while (nxt < nj) {
    const Job jc = mk(cur, par);
    job_cvt(h, r, jc.isw);
    job_load(r, mk(nxt, par), lane);
    unsigned tk;
    draw_ticket(tk, tkt, 1u);
    job_store(h, jc.dst, lane);
    wait_vm<25>();
    st_flag(pflag, call);
    pflag = jc.flag;
    wait_vm<9>();
    asm volatile("" : "+v"(tk) :: "memory");
    pin16(r);
    cur = nxt;
    nxt = (int)(__builtin_amdgcn_readfirstlane(tk) - tbase);
  }
  const Job jc = mk(cur, par);
  job_cvt(h, r, jc.isw);
  job_store(h, jc.dst, lane);
  wait_vm<8>();
  st_flag(pflag, call);
  wait_vm<0>();
  st_flag(jc.flag, call);
}

__device__ __forceinline__ Job x_job(const Params& p, char* ws, int unit, int chunk, unsigned par) {
  Job j;
  j.src = (const char*)p.x + (size_t)unit * 16 * (KK * 4) + chunk * 256 * 4;
  j.dst = ws + par * PAR_B + X_OFF + (unsigned)(p.rank * 16 + unit) * XU_B + chunk * JOB_B;
  j.flag = ws + FX_OFF + ((par * 128 + p.rank * 16 + unit) * 8 + chunk) * FL_STR;
  j.isw = false;
  return j;
}
__device__ __forceinline__ Job w_job(const Params& p, char* ws, int slice, int slot, int pc, unsigned par) {
  Job j;
  j.src = (const char*)p.w + (size_t)pc * 64 * (SH * 4) + slice * 64 * 4;
  j.dst = ws + par * PAR_B + W_OFF + (unsigned)slot * WS_B + pc * JOB_B;
  j.flag = ws + FW_OFF + ((par * 8 + slot) * 32 + pc) * FL_STR;
  j.isw = true;
  return j;
}

__device__ __forceinline__ bool wave_wait(const char* fl, int nfl, unsigned call, unsigned* tmo, int lane) {
  const unsigned* a = (lane < nfl) ? (const unsigned*)(fl + lane * FL_STR) : tmo;
  const unsigned long long t0 = __builtin_amdgcn_s_memrealtime();
  for (;;) {
    const unsigned v = __hip_atomic_load(a, __ATOMIC_RELAXED, __HIP_MEMORY_SCOPE_SYSTEM);
    if (__ballot(lane >= nfl && v == call) != 0ull) return false;
    if (__ballot(lane < nfl && v < call) == 0ull) {
      asm volatile("" ::: "memory");
      return true;
    }
    if (__builtin_amdgcn_s_memrealtime() - t0 > DEADLINE) {
      if (lane == 0) __hip_atomic_store(tmo, call, __ATOMIC_RELAXED, __HIP_MEMORY_SCOPE_AGENT);
      return false;
    }
    __builtin_amdgcn_s_sleep(1);
  }
}

__global__ void __launch_bounds__(NTHREADS, 2) ag_gemm_kernel(Params p) {
  __shared__ __attribute__((aligned(16))) float lds[2 * 8 * 1024];
  const int lane = threadIdx.x & 63;
  const int wave = __builtin_amdgcn_readfirstlane(threadIdx.x >> 6);
  const int b = blockIdx.x, xcd = b & 7, bi = b >> 3;
  const int rank = p.rank, grp = rank & 4, idx = rank & 3;
  char* const myws = p.myws;
  unsigned* const ctr = (unsigned*)(myws + CT_OFF) + b * 8 + wave;
  unsigned* const tmo = (unsigned*)(myws + TM_OFF);
  STAMP_INIT();
  STAMP(0);

  if (bi < 16) {
    const int sid = b;
    const bool is_link = wave == 0 && sid < 7 * WPL;
    if (is_link || wave == 1 || (wave == 2 && sid < NJ_P)) {
      const int link = sid % 7, i = sid / 7;
      const bool cross = link >= 3;
      const int ldst = cross ? (grp ^ 4) + ((idx + link - 3) & 3) : grp + ((idx + 1 + link) & 3);
      const int dst = is_link ? ldst : rank;
      char* const ws = p.ws[dst & 7];
      const int nwj = is_link ? (cross ? 32 : 0) : (wave == 1 ? 128 : 0);
      const int slice0 = is_link ? (dst & 3) : 0, slot0 = is_link ? 4 + idx : 0;
      const int u0 = (is_link && cross) ? NP - 4 : 0;
      const int nj = is_link ? (cross ? NJ_X : NJ_P) : 0;
      run_jobs(is_link ? i : sid, WPL + i, nj, myws + TK_OFF + link * FL_STR,
               [&](int q, unsigned par) {
                 return q < nwj ? w_job(p, ws, slice0 + (q >> 5), slot0 + (q >> 5), q & 31, par)
                                : x_job(p, ws, u0 + (q >> 3), q & 7, par);
               }, lane, wave, ctr);
    } else if (wave == 3 && sid == 127) {
      const unsigned call = __builtin_amdgcn_readfirstlane(
                 __hip_atomic_load(ctr, __ATOMIC_RELAXED, __HIP_MEMORY_SCOPE_AGENT)) + 1u;
      if (lane == 0) __hip_atomic_store(ctr, call, __ATOMIC_RELAXED, __HIP_MEMORY_SCOPE_AGENT);
      const unsigned* a = (const unsigned*)(myws + FD_OFF + lane * FL_STR);
      const unsigned long long t0 = __builtin_amdgcn_s_memrealtime();
      for (;;) {
        const unsigned v = __hip_atomic_load(a, __ATOMIC_RELAXED, __HIP_MEMORY_SCOPE_SYSTEM);
        if (__ballot(v < call) == 0ull) break;
        if (__hip_atomic_load(tmo, __ATOMIC_RELAXED, __HIP_MEMORY_SCOPE_AGENT) == call ||
            __builtin_amdgcn_s_memrealtime() - t0 > DEADLINE) {
          if (lane == 0) {
            __hip_atomic_store(tmo, call, __ATOMIC_RELAXED, __HIP_MEMORY_SCOPE_AGENT);
            p.out[rank][(size_t)rank * SH * SH] = __builtin_nanf("");
          }
          break;
        }
        __builtin_amdgcn_s_sleep(1);
      }
    }
    STAMP(1);
    STAMP_FLUSH();
    return;
  }

  const unsigned call = __builtin_amdgcn_readfirstlane(
                            __hip_atomic_load(ctr, __ATOMIC_RELAXED, __HIP_MEMORY_SCOPE_AGENT)) + 1u;
  if (lane == 0) __hip_atomic_store(ctr, call, __ATOMIC_RELAXED, __HIP_MEMORY_SCOPE_AGENT);
  const unsigned par = call & 1u;
  const int ci = bi - 16;
  const int g = xcd * 2 + (ci >> 3), slot = ci & 7;
  const int odst = slot < 4 ? rank : (grp ^ 4) + (slot - 4);
  float* const obase = p.out[odst & 7] + (slot < 4 ? slot : idx) * 64;
  const unsigned units = p.tab[g];
  const __amdgpu_buffer_rsrc_t rs = __builtin_amdgcn_make_buffer_rsrc(myws, 0, (int)(2 * PAR_B), 0x00020000);
  const int voff = lane * 16;

  h8 B[8][4];
  bool ok = wave_wait(myws + FW_OFF + ((par * 8 + slot) * 32 + wave * 4) * FL_STR, 4, call, tmo, lane);
  STAMP(2);
  {
    const int sb = par * PAR_B + W_OFF + slot * WS_B + wave * 32768;
#pragma unroll
    for (int s = 0; s < 8; ++s)
#pragma unroll
      for (int e = 0; e < 4; ++e)
        B[s][e] = __builtin_bit_cast(h8, __builtin_amdgcn_raw_buffer_load_b128(rs, voff + e * 1024, sb + s * 4096, 17));
  }

#pragma unroll 1
  for (int t = 0; t < 4; ++t) {
    const int su = (units >> (8 * t)) & 127;
    ok = wave_wait(myws + FX_OFF + ((par * 128 + su) * 8 + wave) * FL_STR, 1, call, tmo, lane) && ok;
    STAMP(3 + 4 * t);
    const int sb = par * PAR_B + X_OFF + su * XU_B + wave * JOB_B;
    h8 A[8];
#pragma unroll
    for (int s = 0; s < 8; ++s)
      A[s] = __builtin_bit_cast(h8, __builtin_amdgcn_raw_buffer_load_b128(rs, voff + (s & 3) * 1024, sb + (s >> 2) * 4096, 17));
    __builtin_amdgcn_sched_barrier(0);
    f4 acc[4];
#pragma unroll
    for (int e = 0; e < 4; ++e) acc[e] = f4{0.f, 0.f, 0.f, 0.f};
#pragma unroll
    for (int s = 0; s < 8; ++s)
#pragma unroll
      for (int e = 0; e < 4; ++e)
        acc[e] = __builtin_amdgcn_mfma_f32_16x16x32_f16(A[s], B[s][e], acc[e], 0, 0, 0);
    if (!ok) {
      const float nan = __builtin_nanf("");
#pragma unroll
      for (int e = 0; e < 4; ++e) acc[e] = f4{nan, nan, nan, nan};
    }
    float* const buf = lds + (t & 1) * 8192;
    {
      float* slab = buf + wave * 1024 + (lane >> 4) * 4 * 64 + (lane & 15) * 4;
#pragma unroll
      for (int r = 0; r < 4; ++r) *(f4*)(slab + r * 64) = f4{acc[0][r], acc[1][r], acc[2][r], acc[3][r]};
    }
    STAMP(4 + 4 * t);
    __syncthreads();
    STAMP(5 + 4 * t);
    if ((wave >> 2) == (t & 1)) {
      const int i = threadIdx.x & 255;
      f4 s = *(const f4*)(buf + i * 4);
#pragma unroll
      for (int v = 1; v < 8; ++v) s += *(const f4*)(buf + v * 1024 + i * 4);
#pragma unroll
      for (int j = 0; j < 4; ++j) s[j] = s[j] < 0.f ? 0.f : s[j];
      float* o = obase + (size_t)(su * 16 + (i >> 4)) * SH + (i & 15) * 4;
      st16_sys<0>((char*)o, __builtin_bit_cast(u4, s));
    }
    STAMP(6 + 4 * t);
  }
  if (slot >= 4) {
    wait_vm<0>();
    __syncthreads();
    if (threadIdx.x == 0)
      __hip_atomic_store((unsigned*)(p.ws[odst & 7] + FD_OFF + (idx * 16 + g) * FL_STR), call,
                         __ATOMIC_RELAXED, __HIP_MEMORY_SCOPE_SYSTEM);
  }
  STAMP(1);
  STAMP_FLUSH();
}

}

extern "C" void kernel_launch(int rank, int nranks, ncclComm_t comm, hipStream_t stream,
                              void* const* d_in, const int* in_sizes, int n_in,
                              void* d_out, int out_size,
                              const int* in_dtypes, int out_dtype,
                              void* const* d_out_all,
                              void* const* d_ws_all, size_t ws_bytes) {
  (void)comm; (void)in_dtypes; (void)out_dtype; (void)d_out;
  if (nranks != NR || (unsigned)rank >= (unsigned)NR || n_in != 2 || in_sizes[0] != SH * KK ||
      in_sizes[1] != KK * SH || out_size != MM * SH || ws_bytes < WS_END) {
    fprintf(stderr, "unexpected problem shape\n");
    return;
  }
  Params p;
  p.x = (const float*)d_in[0];
  p.w = (const float*)d_in[1];
  for (int i = 0; i < NR; ++i) { p.ws[i] = (char*)d_ws_all[i]; p.out[i] = (float*)d_out_all[i]; }
  p.myws = p.ws[rank];
  p.rank = rank;
  p.pad = 0;
  int ent[64], key[64], n = 0;
  for (int s = 0; s < NR; ++s) {
    const bool same = ((s ^ rank) & 4) == 0;
    for (int u = same ? 0 : NP; u < (same ? NP : 16); ++u) {
      const int k = s == rank ? 0 : (same ? 10 * (u + 1) : 53 + 14 * (u - NP));
      int j = n++;
      for (; j > 0 && key[j - 1] > k; --j) { key[j] = key[j - 1]; ent[j] = ent[j - 1]; }
      key[j] = k; ent[j] = s * 16 + u;
    }
  }
  for (int g = 0; g < 16; ++g)
    p.tab[g] = (unsigned)ent[g] | (unsigned)ent[16 + g] << 8 | (unsigned)ent[32 + g] << 16 | (unsigned)ent[48 + g] << 24;
  hipLaunchKernelGGL(ag_gemm_kernel, dim3(NBLOCKS), dim3(NTHREADS), 0, stream, p);
  HIP_CHECK(hipGetLastError());
}
```

```cpp
#include <hip/hip_runtime.h>
#include <rccl/rccl.h>
#include <cstdio>
#include <cstdint>

#define HIP_CHECK(expr) do { hipError_t _e = (expr); if (_e != hipSuccess) { \
    fprintf(stderr, "HIP error %s at %s:%d\n", hipGetErrorString(_e), __FILE__, __LINE__); \
    return; } } while (0)

#define WPLP 8
#define WPLX 8
#define WIN 6
#define S0 6
#define LA 3
#define LOCDELAY 24
#define ZT0 300
#define ZMODE 0
#define ZWB 0
#define LOCPLAIN 0
#define ZSTEP 16
#define LDSTAG 8
#define OFULL 1550
#define REL0 1150
#define RELSTEP 28
#define RELROT 0
#define OPOLL0 1000
#define PROBE 4096
#define PRED 0
#define SLROT 1
#define SLOTX 0
#define PSLEEP 1
#define NP 11

namespace {

typedef _Float16 h8 __attribute__((ext_vector_type(8)));
typedef float f4 __attribute__((ext_vector_type(4)));
typedef unsigned u4 __attribute__((ext_vector_type(4)));
typedef unsigned long long u64;

constexpr int NR = 8;
constexpr int MM = 2048, KK = 2048;
constexpr int SH = 256;
constexpr int NTHREADS = 512, NBLOCKS = 256;
constexpr int NJ_P = NP * 8;
constexpr int NJ_X = 32 + (16 - NP) * 8;

constexpr int NS_P = 16, NS_X = 16;
static_assert(S0 >= 4 && S0 <= 8 && S0 * 8 <= NJ_P && S0 * 8 <= NJ_X && NJ_P <= 8 * NS_P && NJ_X <= 8 * NS_X && S0 < NS_X, "");

constexpr unsigned XU_B = 16 * KK * 2;
constexpr unsigned WS_B = 64 * KK * 2;
constexpr unsigned JOB_B = 8192;
constexpr unsigned XB_B = NR * 16 * XU_B;
constexpr unsigned WB_B = 8 * WS_B;
constexpr unsigned X_OFF = 0;
constexpr unsigned W_OFF = 3 * XB_B;
constexpr unsigned FL_STR = 128;
constexpr unsigned FW_OFF = W_OFF + 3 * WB_B;
constexpr unsigned MK_OFF = FW_OFF + 4 * 32 * FL_STR;
constexpr unsigned TK_OFF = MK_OFF + 4 * FL_STR;
constexpr unsigned TM_OFF = TK_OFF + 8 * FL_STR;
constexpr unsigned CT_OFF = TM_OFF + FL_STR;
constexpr unsigned ST_OFF = CT_OFF + NBLOCKS * 8 * 4;
constexpr unsigned WS_END = ST_OFF + NBLOCKS * 8 * 32 * 8;
constexpr u64 DEADLINE = 100000000ull;
constexpr unsigned MARK = 0xBF800000u;

struct Params {
  const float* x;
  const float* w;
  char* myws;
  int rank;
  int pad;
  char* ws[NR];
  float* out[NR];
  unsigned tab[16];
  unsigned otab[16];
};

#define STAMPV(i, v) do {} while (0)
#define STAMP(i) do {} while (0)
#define STAMP_INIT() do {} while (0)
#define STAMP_FLUSH() do {} while (0)

template <int OFF>
__device__ __forceinline__ void st16_sys(char* a, u4 v) {
  asm volatile("global_store_dwordx4 %0, %1, off offset:%2 sc0 sc1\n\ts_nop 1"
               :: "v"(a), "v"(v), "i"(OFF) : "memory");
}
__device__ __forceinline__ const char* sgpr_ptr(const char* a) {
  const unsigned lo = __builtin_amdgcn_readfirstlane((unsigned)(u64)a), hi = __builtin_amdgcn_readfirstlane((unsigned)((u64)a >> 32));
  const char* r = (const char*)((u64)lo | (u64)hi << 32);
  asm volatile("s_nop 4" : "+s"(r));
  return r;
}
__device__ __forceinline__ void load16(f4 (&r)[16], const char* src, unsigned vo, unsigned sa, unsigned sb, unsigned sc) {
  const char* a = sgpr_ptr(src);
#pragma unroll
  for (int k = 0; k < 16; ++k)
    asm volatile("global_load_dwordx4 %0, %1, %2" : "=v"(r[k])
                 : "v"(vo + (k >> 3) * sa + ((k >> 1) & 3) * sb + (k & 1) * sc), "s"(a) : "memory");
}
__device__ __forceinline__ void st16_sys_s(const char* base, unsigned vo, u4 v) {
  asm volatile("global_store_dwordx4 %0, %1, %2 sc0 sc1\n\ts_nop 1" :: "v"(vo), "v"(v), "s"(base) : "memory");
}
__device__ __forceinline__ void st_flag(char* a, unsigned v) {
  asm volatile("s_mov_b64 exec, 1\n\tglobal_store_dword %0, %1, off sc0 sc1\n\ts_mov_b64 exec, -1"
               :: "v"(a), "v"(v) : "memory");
}
#define RAW_BARRIER() do { asm volatile("s_waitcnt lgkmcnt(0)" ::: "memory"); __builtin_amdgcn_s_barrier(); asm volatile("" ::: "memory"); } while (0)
template <int N>
__device__ __forceinline__ void wait_vm() { asm volatile("s_waitcnt vmcnt(%0)" :: "i"(N) : "memory"); }
__device__ __forceinline__ unsigned next_call(unsigned* ctr, int lane) {
  const unsigned call = __builtin_amdgcn_readfirstlane(
                            __hip_atomic_load(ctr, __ATOMIC_RELAXED, __HIP_MEMORY_SCOPE_AGENT)) + 1u;
  if (lane == 0) __hip_atomic_store(ctr, call, __ATOMIC_RELAXED, __HIP_MEMORY_SCOPE_AGENT);
  return call;
}

struct Job {
  const char* src;
  char* dst;
  char* flag;
  bool isw;
};

__device__ __forceinline__ void job_load(f4 (&r)[16], const Job& jb, int lane) {
  const unsigned vo = jb.isw ? (lane >> 4) * (8 * SH * 4) + (lane & 15) * 16 : (lane & 15) * (KK * 4) + (lane >> 4) * 32;
  load16(r, jb.src, vo, jb.isw ? 32 * SH * 4 : 512, jb.isw ? 2 * SH * 4 : 128, jb.isw ? SH * 4 : 16);
}
__device__ __forceinline__ void pin16(f4 (&r)[16]) {
  asm volatile("" : "+v"(r[0]), "+v"(r[1]), "+v"(r[2]), "+v"(r[3]), "+v"(r[4]), "+v"(r[5]), "+v"(r[6]),
               "+v"(r[7]), "+v"(r[8]), "+v"(r[9]), "+v"(r[10]), "+v"(r[11]), "+v"(r[12]), "+v"(r[13]),
               "+v"(r[14]), "+v"(r[15]) :: "memory");
}
__device__ __forceinline__ void job_cvt(u4 (&h)[8], const f4 (&r)[16], bool isw) {
  if (!isw) {
#pragma unroll
    for (int f = 0; f < 8; ++f) {
      h8 v;
#pragma unroll
      for (int j = 0; j < 8; ++j) v[j] = (_Float16)r[2 * f + (j >> 2)][j & 3];
      h[f] = __builtin_bit_cast(u4, v);
#pragma unroll
      for (int j = 0; j < 4; ++j) h[f][j] = h[f][j] ? h[f][j] : 0x80008000u;
    }
  } else {
#pragma unroll
    for (int f = 0; f < 8; ++f) {
      h8 v;
#pragma unroll
      for (int j = 0; j < 8; ++j) v[j] = (_Float16)r[(f >> 2) * 8 + j][f & 3];
      h[f] = __builtin_bit_cast(u4, v);
#pragma unroll
      for (int j = 0; j < 4; ++j) h[f][j] = h[f][j] ? h[f][j] : 0x80008000u;
    }
  }
}
__device__ __forceinline__ void job_store(const u4 (&h)[8], char* dst, int lane) {
  char* a = dst + lane * 16 + 4096;
  st16_sys<-4096>(a, h[0]); st16_sys<-3072>(a, h[1]); st16_sys<-2048>(a, h[2]); st16_sys<-1024>(a, h[3]);
  st16_sys<0>(a, h[4]);     st16_sys<1024>(a, h[5]);  st16_sys<2048>(a, h[6]);  st16_sys<3072>(a, h[7]);
}

__device__ __forceinline__ Job x_job(const Params& p, char* ws, int unit, int chunk, unsigned call) {
  Job j;
  j.src = (const char*)p.x + (size_t)unit * 16 * (KK * 4) + chunk * 256 * 4;
  j.dst = ws + X_OFF + (call % 3u) * XB_B + (unsigned)(p.rank * 16 + unit) * XU_B + chunk * JOB_B;
  j.flag = nullptr;
  j.isw = false;
  return j;
}
__device__ __forceinline__ Job w_job(const Params& p, char* ws, int slice, int slot, int pc, unsigned call) {
  Job j;
  j.src = (const char*)p.w + (size_t)pc * 64 * (SH * 4) + slice * 64 * 4;
  j.dst = ws + W_OFF + (call % 3u) * WB_B + (unsigned)slot * WS_B + pc * JOB_B;
  j.flag = ws + FW_OFF + ((slot & 3) * 32 + pc) * FL_STR;
  j.isw = true;
  return j;
}

__global__ void __launch_bounds__(NTHREADS, 2) ag_gemm_kernel(Params p) {
  __shared__ __attribute__((aligned(16))) float lds[4 * 8 * 1024 + 32];
  const int lane = threadIdx.x & 63;
  const int wave = __builtin_amdgcn_readfirstlane(threadIdx.x >> 6);
  const int b = blockIdx.x, xcd = b & 7, bi = b >> 3;
  const int rank = p.rank, grp = rank & 4, idx = rank & 3;
  char* const myws = p.myws;
  unsigned* const ctr = (unsigned*)(myws + CT_OFF) + b * 8 + wave;
  unsigned* const tmo = (unsigned*)(myws + TM_OFF);
  const u64 tstart = __builtin_amdgcn_s_memrealtime();
  STAMP_INIT();
  STAMP(0);

  if (bi < 16) {
    const int sid = b;
    typedef __attribute__((address_space(3))) unsigned* lds_u32_p;
    typedef __attribute__((address_space(3))) u4* lds_u4_p;
    if (sid < 56) {
      const int link = sid >> 3, i = sid & 7;
      const bool cross = link >= 3;
      const int dst = cross ? (grp ^ 4) + ((idx + link - 3) & 3) : grp + ((idx + 1 + link) & 3);
      char* const ws = p.ws[dst & 7];
      const int nj = cross ? NJ_X : NJ_P, n = cross ? NS_X : NS_P;
      auto mk = [&](int q, unsigned c) {
        return cross && q < 32 ? w_job(p, ws, (dst + SLROT * rank) & 3, 4 + idx, q, c)
                               : x_job(p, ws, (cross ? NP - 4 : 0) + (q >> 3), q & 7, c);
      };
      typedef __attribute__((address_space(3))) char* lds_p;
      typedef unsigned u2 __attribute__((ext_vector_type(2)));
      const lds_p ring = (lds_p)lds;
      const int sl0 = ((lane & 48) + ((lane & 15) ^ (lane >> 4))) * 16, sl1 = sl0 ^ 64;
      const lds_u32_p full = (lds_u32_p)(lds + 4 * 8192);
      unsigned c;
      asm volatile("global_load_dword %0, %1, off sc1" : "=v"(c) : "v"(ctr) : "memory");
      if (wave == 0) {
        if (lane < 32) full[lane] = 0u;
      } else {
        RAW_BARRIER();
        if (LDSTAG) for (int k = 1; k < wave; ++k) __builtin_amdgcn_s_sleep(LDSTAG);
      }
      unsigned call = 0;
#pragma unroll 1
      for (int j = wave < S0 ? wave : n; j < n; j = wave > 3 ? n : j < S0 ? S0 + wave - 1 : j + 3) {
        int q = i + 8 * j;
        if (j >= S0) {
          while ((int)__hip_atomic_load(full + 25, __ATOMIC_RELAXED, __HIP_MEMORY_SCOPE_WORKGROUP) < j - LA ||
                 (int)__hip_atomic_load(full + 26, __ATOMIC_RELAXED, __HIP_MEMORY_SCOPE_WORKGROUP) < j - S0)
            __builtin_amdgcn_s_sleep(2);
          if (__hip_atomic_load(full + 26, __ATOMIC_RELAXED, __HIP_MEMORY_SCOPE_WORKGROUP) >= 1000u) break;
          unsigned t = 0;
          if (lane == 0)
            t = __hip_atomic_fetch_add((unsigned*)(myws + TK_OFF + link * FL_STR), 1u, __ATOMIC_RELAXED,
                                       __HIP_MEMORY_SCOPE_AGENT);
          q = (int)(__builtin_amdgcn_readfirstlane(t) - (call - 1u) * (unsigned)(8 * (n - S0))) + 8 * S0;
          if (lane == 0)
            __hip_atomic_store(full + 26, q < nj ? (unsigned)(j + 1 - S0) : 1000u, __ATOMIC_RELAXED, __HIP_MEMORY_SCOPE_WORKGROUP);
          if (q >= nj) {
            if (lane == 0) {
              __hip_atomic_store(full + j, 1000u, __ATOMIC_RELAXED, __HIP_MEMORY_SCOPE_WORKGROUP);
              __hip_atomic_fetch_add((unsigned*)(myws + TK_OFF + link * FL_STR), (unsigned)(n - 1 - j), __ATOMIC_RELAXED,
                                     __HIP_MEMORY_SCOPE_AGENT);
            }
            break;
          }
        }
        const bool isw = cross && q < 32;
        f4 r[16];
        load16(r, mk(q, 0u).src, isw ? (lane >> 4) * (8 * SH * 4) + (lane & 15) * 16 : lane * 16,
               isw ? 32 * SH * 4 : 8 * KK * 4, isw ? 2 * SH * 4 : 2 * KK * 4, isw ? SH * 4 : KK * 4);
        if (wave == 0) RAW_BARRIER();
        wait_vm<0>();
        pin16(r);
        asm volatile("" : "+v"(c) :: "memory");
        call = __builtin_amdgcn_readfirstlane(c) + 1u;
        const lds_p img = ring + j * JOB_B;
        if (isw) {
          u4 h[8];
          job_cvt(h, r, true);
#pragma unroll
          for (int f = 0; f < 8; ++f) *(lds_u4_p)(img + f * 1024 + ((f & 1) ? sl1 : sl0)) = h[f];
        } else {
          typedef _Float16 h4 __attribute__((ext_vector_type(4)));
          const int g = (lane >> 1) & 3, sw = ((lane >> 1) & 7);
          const lds_p a = img + (lane >> 3) * 1024 + g * 256 + (lane & 1) * 8;
#pragma unroll
          for (int k = 0; k < 16; ++k) {
            const h4 v = {(_Float16)r[k][0], (_Float16)r[k][1], (_Float16)r[k][2], (_Float16)r[k][3]};
            u2 w2 = __builtin_bit_cast(u2, v);
            w2[0] = w2[0] ? w2[0] : 0x80008000u;
            w2[1] = w2[1] ? w2[1] : 0x80008000u;
            *(__attribute__((address_space(3))) u2*)(a + (k ^ sw) * 16) = w2;
          }
        }
        if (wave == 0) break;
        if (cross && q == 31) {
          const unsigned* m = (const unsigned*)(myws + MK_OFF + (link - 3) * FL_STR);
          while (__hip_atomic_load(m, __ATOMIC_RELAXED, __HIP_MEMORY_SCOPE_AGENT) - (call - 1u) * 64u < 64u) {
            if (__builtin_amdgcn_s_memrealtime() - tstart > DEADLINE) {
              if (lane == 0) __hip_atomic_store(tmo, call, __ATOMIC_RELAXED, __HIP_MEMORY_SCOPE_AGENT);
              const u4 nan = {0x7E007E00u, 0x7E007E00u, 0x7E007E00u, 0x7E007E00u};
#pragma unroll
              for (int f = 0; f < 8; ++f) *(lds_u4_p)(img + f * 1024 + sl0) = nan;
              break;
            }
            __builtin_amdgcn_s_sleep(2);
          }
        }
        __builtin_amdgcn_fence(__ATOMIC_RELEASE, "workgroup");
        if (lane == 0) __hip_atomic_store(full + j, (unsigned)q + 1u, __ATOMIC_RELAXED, __HIP_MEMORY_SCOPE_WORKGROUP);
        STAMP(j < S0 ? 2 : 3);
      }
      if (wave >= S0) {
        wait_vm<0>();
        asm volatile("" : "+v"(c) :: "memory");
        call = __builtin_amdgcn_readfirstlane(c) + 1u;
      }
      if (wave != 0 && lane == 0) __hip_atomic_store(ctr, call, __ATOMIC_RELAXED, __HIP_MEMORY_SCOPE_AGENT);
      if (wave == 0) {
        STAMP(2);
#pragma unroll 1
        for (int j = 0; j < n; ++j) {
          unsigned e = i + 1;
          if (j) while ((e = __hip_atomic_load(full + j, __ATOMIC_ACQUIRE, __HIP_MEMORY_SCOPE_WORKGROUP)) == 0u)
            __builtin_amdgcn_s_sleep(1);
          const int q = (int)__builtin_amdgcn_readfirstlane(e) - 1;
          if (q >= nj) break;
          u4 h[8];
#pragma unroll
          for (int f = 0; f < 8; ++f) h[f] = *(lds_u4_p)(ring + j * JOB_B + f * 1024 + ((f & 1) ? sl1 : sl0));
          wait_vm<WIN>();
          job_store(h, mk(q, call).dst, lane);
          if (lane == 0) __hip_atomic_store(full + 25, (unsigned)j + 1u, __ATOMIC_RELAXED, __HIP_MEMORY_SCOPE_WORKGROUP);
        }
        if (lane == 0) __hip_atomic_store(full + 25, 1000u, __ATOMIC_RELAXED, __HIP_MEMORY_SCOPE_WORKGROUP);
        st_flag((char*)ctr, call);
        STAMP(28);
      }
    } else if (sid >= 64) {
      const int hid = sid - 64;
      const unsigned call = next_call(ctr, lane);
      if (LOCDELAY) __builtin_amdgcn_s_sleep(LOCDELAY);
      if (wave < 4) {
        const int q = hid + 64 * (wave & 1);
        if (wave < 2 || q < NJ_P) {
          const Job jb = wave < 2 ? w_job(p, myws, q >> 5, q >> 5, q & 31, call) : x_job(p, myws, q >> 3, q & 7, call);
          f4 r[16];
          u4 h[8];
          job_load(r, jb, lane);
          wait_vm<0>();
          pin16(r);
          job_cvt(h, r, jb.isw);
          job_store(h, jb.dst, lane);
          if (jb.isw) {
            wait_vm<0>();
            st_flag(jb.flag, call);
          }
        }
        STAMP(2);
        while (__builtin_amdgcn_s_memrealtime() - tstart < ZT0) __builtin_amdgcn_s_sleep(8);
        const int su = (p.tab[hid & 15] >> (8 * (hid >> 4))) & 127;
        char* a = myws + X_OFF + ((call + 2u) % 3u) * XB_B + su * XU_B + wave * (XU_B / 4) + lane * 16;
        const u4 z = {0u, 0u, 0u, 0u};
#pragma unroll 1
        for (int j = 0; j < 20; ++j) {
          if (j == 16) a = myws + W_OFF + ((call + 2u) % 3u) * WB_B + (4 + wave) * WS_B + hid * 4096 - 16384 + lane * 16;
          st16_sys<0>(a + j * 1024, z);
          if (ZSTEP) __builtin_amdgcn_s_sleep(ZSTEP);
        }
        if (ZWB && wave == 0 && hid < 8) {
          while (__builtin_amdgcn_s_memrealtime() - tstart < ZWB) __builtin_amdgcn_s_sleep(8);
          __builtin_amdgcn_fence(__ATOMIC_RELEASE, "agent");
          wait_vm<0>();
        }
      } else {
        const int col = ((idx + wave + SLROT * rank) & 3) * 64;
        const int lo = ((lane >> 4) * SH + col + (lane & 15) * 4) * 4;
        const int o0 = (int)((p.otab[hid >> 2] >> (8 * (hid & 3))) & 127) * (16 * SH * 4);
        char* const out = (char*)p.out[rank];
        const u4 mk = {MARK, MARK, MARK, MARK};
#pragma unroll
        for (int j = 0; j < 4; ++j) st16_sys<0>(out + o0 + lo + j * 4 * SH * 4, mk);
        wait_vm<0>();
        if (lane == 0)
          __hip_atomic_fetch_add((unsigned*)(myws + MK_OFF + (wave - 4) * FL_STR), 1u, __ATOMIC_RELAXED,
                                 __HIP_MEMORY_SCOPE_AGENT);
        STAMP(29);
        while (__builtin_amdgcn_s_memrealtime() - tstart < OPOLL0) __builtin_amdgcn_s_sleep(8);
        const __amdgpu_buffer_rsrc_t ro = __builtin_amdgcn_make_buffer_rsrc(out, 0, MM * SH * 4, 0x00020000);
        const unsigned* const one = lane == 0 ? (const unsigned*)(out + o0 + (15 * SH + col + 63) * 4) : tmo;
        bool full = false;
        for (;;) {
          bool left = true;
          if (full) {
            u4 v[4];
#pragma unroll
            for (int j = 0; j < 4; ++j) v[j] = __builtin_amdgcn_raw_buffer_load_b128(ro, lo + j * 4 * SH * 4, o0, 17);
            left = false;
#pragma unroll
            for (int j = 0; j < 4; ++j)
#pragma unroll
              for (int k = 0; k < 4; ++k) left |= v[j][k] == MARK;
          }
          const unsigned v1 = __hip_atomic_load(one, __ATOMIC_RELAXED, __HIP_MEMORY_SCOPE_SYSTEM);
          if (__ballot(left) == 0ull) break;
          const u64 el = __builtin_amdgcn_s_memrealtime() - tstart;
          if (__ballot(lane != 0 && v1 == call) != 0ull || el > DEADLINE) {
            if (lane == 0) {
              __hip_atomic_store(tmo, call, __ATOMIC_RELAXED, __HIP_MEMORY_SCOPE_AGENT);
              p.out[rank][(size_t)rank * SH * SH] = __builtin_nanf("");
            }
            break;
          }
          full = el > OFULL || __builtin_amdgcn_readfirstlane(v1) != MARK;
          asm volatile("" ::: "memory");
          if (!full) __builtin_amdgcn_s_sleep(4);
        }
      }
    }
    STAMP(1);
    STAMP_FLUSH();
    return;
  }

  const unsigned call = next_call(ctr, lane);
  const int ci = bi - 16;
  const int g = SLOTX ? ci : xcd * 2 + (ci >> 3), slot = SLOTX ? xcd : ci & 7;
  const int odst = slot < 4 ? rank : (grp ^ 4) + (slot - 4);
  float* const obase = p.out[odst & 7] + (slot < 4 ? slot : (idx + SLROT * odst) & 3) * 64;
  const unsigned units = p.tab[g];
  const __amdgpu_buffer_rsrc_t rs = __builtin_amdgcn_make_buffer_rsrc(myws, 0, (int)FW_OFF, 0x00020000);
  const int voff = lane * 16;

  typedef __attribute__((address_space(3))) unsigned* lds_u32_p;
  const lds_u32_p cnt = (lds_u32_p)(lds + 4 * 8192);
  if (threadIdx.x < 8) cnt[threadIdx.x] = 0u;
  __syncthreads();

  const bool remote = slot >= 4;
  h8 B[8][4];
  bool ok = true;
  {
    const int sb = W_OFF + (call % 3u) * WB_B + slot * WS_B + wave * 32768;
    const unsigned* a = (const unsigned*)(myws + (lane >= 4 ? TM_OFF : remote ? sb + lane * JOB_B + PROBE
                                                  : FW_OFF + (slot * 32 + wave * 4 + lane) * FL_STR));
    const unsigned least = remote ? 1u : call;
#pragma unroll 1
    for (;;) {
      const unsigned v = __hip_atomic_load(a, __ATOMIC_RELAXED, __HIP_MEMORY_SCOPE_SYSTEM);
      if (__ballot(lane >= 4 && v == call) != 0ull) ok = false;
      if (__builtin_amdgcn_s_memrealtime() - tstart > DEADLINE) {
        if (lane == 0) __hip_atomic_store(tmo, call, __ATOMIC_RELAXED, __HIP_MEMORY_SCOPE_AGENT);
        ok = false;
      }
      if (ok && __ballot(lane < 4 && v < least) != 0ull) {
        __builtin_amdgcn_s_sleep(PSLEEP);
        continue;
      }
      asm volatile("" ::: "memory");
      STAMP(2);
      unsigned lo = ~0u;
#pragma unroll
      for (int s = 0; s < 8; ++s)
#pragma unroll
        for (int e = 0; e < 4; ++e) {
          const u4 v4 = __builtin_amdgcn_raw_buffer_load_b128(rs, voff + e * 1024, sb + s * 4096, 17);
          B[s][e] = __builtin_bit_cast(h8, v4);
        }
      __builtin_amdgcn_sched_barrier(0);
#pragma unroll
      for (int s = 0; s < 8; ++s)
#pragma unroll
        for (int e = 0; e < 4; ++e) {
          const u4 v4 = __builtin_bit_cast(u4, B[s][e]);
          lo = min(min(lo, min(v4[0], v4[1])), min(v4[2], v4[3]));
        }
      if (!ok || __ballot(lo == 0u) == 0ull) break;
    }
    ok = __builtin_amdgcn_readfirstlane(ok) != 0;
    STAMP(19);
  }

  const unsigned xb = X_OFF + (call % 3u) * XB_B + wave * JOB_B;
  const unsigned lu = (units >> (8 * (lane & 3))) & 127;
  const bool early = lane == 3 && (wave >= PRED || ((lu & 15) != 0 && (lu & 15) != NP));
  const unsigned* const probe = (const unsigned*)(myws +
      (lane < 4 ? xb + lu * XU_B + PROBE - (early ? PRED * JOB_B : 0) : TM_OFF));
  const unsigned release = remote ? REL0 + RELSTEP * (15 - ((g + RELROT * slot) & 15)) : 0u;
  const unsigned vo = (voff & 0x300) * 4 + (voff & 0xF0);
  unsigned todo = 15u, rdy = 0u;
  u64 now = tstart;
#pragma unroll 1
  for (;;) {
    int red = -1;
    if (!rdy) {
      unsigned v = 0u;
      if (todo) v = __hip_atomic_load(probe, __ATOMIC_RELAXED, __HIP_MEMORY_SCOPE_SYSTEM);
      now = __builtin_amdgcn_s_memrealtime();
      const unsigned sent = __builtin_amdgcn_readfirstlane(__hip_atomic_load(cnt + 4, __ATOMIC_RELAXED, __HIP_MEMORY_SCOPE_WORKGROUP));
      const unsigned held = __builtin_amdgcn_readfirstlane(__hip_atomic_load(cnt + 5, __ATOMIC_RELAXED, __HIP_MEMORY_SCOPE_WORKGROUP));
      const unsigned tosum = __builtin_amdgcn_readfirstlane(__hip_atomic_load(cnt + 6, __ATOMIC_RELAXED, __HIP_MEMORY_SCOPE_WORKGROUP));
      rdy = (unsigned)__ballot(v != 0u) & todo;
      if (__ballot(lane >= 4 && v == call) != 0ull) { ok = false; rdy = todo; }
      if (now - tstart > DEADLINE) {
        if (lane == 0) __hip_atomic_store(tmo, call, __ATOMIC_RELAXED, __HIP_MEMORY_SCOPE_AGENT);
        if (!todo && !tosum && !held) break;
        ok = false;
        rdy = todo;
      }
      if (!rdy) {
        if (tosum) {
          const unsigned bit = tosum & (0u - tosum);
          unsigned old = 0u;
          if (lane == 0) old = __hip_atomic_fetch_and(cnt + 6, ~bit, __ATOMIC_ACQUIRE, __HIP_MEMORY_SCOPE_WORKGROUP);
          if (!(__builtin_amdgcn_readfirstlane(old) & bit)) continue;
          red = __builtin_ctz(bit);
        } else if (held && (unsigned)(now - tstart) >= release) {
          unsigned m = 0u;
          if (lane == 0) m = __hip_atomic_exchange(cnt + 5, 0u, __ATOMIC_ACQUIRE, __HIP_MEMORY_SCOPE_WORKGROUP);
          m = __builtin_amdgcn_readfirstlane(m);
          __builtin_amdgcn_fence(__ATOMIC_ACQUIRE, "workgroup");
#pragma unroll 1
          for (; m; m &= m - 1u) {
            const int t = __builtin_ctz(m);
            const float* buf = lds + t * 8192 + lane * 4;
            const char* o = sgpr_ptr((char*)obase + ((units >> (8 * t)) & 127) * (16 * SH * 4));
            f4 sum[4];
#pragma unroll
            for (int j = 0; j < 4; ++j) sum[j] = *(const f4*)(buf + j * 256);
#pragma unroll
            for (int j = 0; j < 4; ++j) st16_sys_s(o, vo + j * 4 * SH * 4, __builtin_bit_cast(u4, sum[j]));
            if (lane == 0) __hip_atomic_fetch_add(cnt + 4, 1u, __ATOMIC_RELAXED, __HIP_MEMORY_SCOPE_WORKGROUP);
            STAMPV(23 + t, 2);
          }
          continue;
        } else {
          if (!todo && sent == 4u) break;
          __builtin_amdgcn_s_sleep(PSLEEP);
          continue;
        }
      }
    }
    if (rdy) {
      const int t = __builtin_ctz(rdy);
      STAMPV(3 + 4 * it, t);
      const int sb = __builtin_amdgcn_readfirstlane(xb + ((units >> (8 * t)) & 127) * XU_B);
      asm volatile("" ::: "memory");
      u4 A[8];
#pragma unroll
      for (int s = 0; s < 8; ++s)
        A[s] = __builtin_amdgcn_raw_buffer_load_b128(rs, voff + (s & 3) * 1024, sb + (s >> 2) * 4096, 17);
      __builtin_amdgcn_sched_barrier(0);
      f4 acc[4];
#pragma unroll
      for (int e = 0; e < 4; ++e) acc[e] = f4{0.f, 0.f, 0.f, 0.f};
      unsigned lo = ~0u;
#pragma unroll
      for (int s = 0; s < 8; ++s) {
#pragma unroll
        for (int e = 0; e < 4; ++e)
          acc[e] = __builtin_amdgcn_mfma_f32_16x16x32_f16(__builtin_bit_cast(h8, A[s]), B[s][e], acc[e], 0, 0, 0);
        lo = min(min(lo, min(A[s][0], A[s][1])), min(A[s][2], A[s][3]));
      }
      const bool missing = __ballot(lo == 0u) != 0ull;
      if (ok & missing) {
        if (__builtin_amdgcn_s_memrealtime() - tstart <= DEADLINE) continue;
        if (lane == 0) __hip_atomic_store(tmo, call, __ATOMIC_RELAXED, __HIP_MEMORY_SCOPE_AGENT);
        ok = false;
      }
      rdy &= rdy - 1u;
      todo &= ~(1u << t);
      if (!ok) {
        const float nan = __builtin_nanf("");
#pragma unroll
        for (int e = 0; e < 4; ++e) acc[e] = f4{nan, nan, nan, nan};
      }
      float* slab = lds + t * 8192 + wave * 1024 + (lane >> 4) * 4 * 64 + (lane & 15) * 4;
#pragma unroll
      for (int r = 0; r < 4; ++r) *(f4*)(slab + r * 64) = f4{acc[0][r], acc[1][r], acc[2][r], acc[3][r]};
      unsigned old = 0;
      if (lane == 0) old = __hip_atomic_fetch_add(cnt + t, 1u, __ATOMIC_ACQ_REL, __HIP_MEMORY_SCOPE_WORKGROUP);
      STAMP(4 + 4 * it);
      if (__builtin_amdgcn_readfirstlane(old) == 7u) {
        if (todo) {
          if (lane == 0) __hip_atomic_fetch_or(cnt + 6, 1u << t, __ATOMIC_RELEASE, __HIP_MEMORY_SCOPE_WORKGROUP);
        } else {
          red = t;
        }
      }
    }
    if (red >= 0) {
      __builtin_amdgcn_fence(__ATOMIC_ACQUIRE, "workgroup");
      now = __builtin_amdgcn_s_memrealtime();
      float* const buf = lds + red * 8192;
      f4 sum[4], pa[4][4];
#pragma unroll
      for (int hf = 0; hf < 2; ++hf) {
#pragma unroll
        for (int v = 0; v < 4; ++v)
#pragma unroll
          for (int j = 0; j < 4; ++j) pa[v][j] = *(const f4*)(buf + (hf * 4 + v) * 1024 + j * 256 + lane * 4);
        __builtin_amdgcn_sched_barrier(0);
#pragma unroll
        for (int j = 0; j < 4; ++j) {
          const f4 q = (pa[0][j] + pa[1][j]) + (pa[2][j] + pa[3][j]);
          sum[j] = hf ? sum[j] + q : q;
        }
        asm volatile("" : "+v"(sum[0]), "+v"(sum[1]), "+v"(sum[2]), "+v"(sum[3]));
      }
#pragma unroll
      for (int j = 0; j < 4; ++j)
#pragma unroll
        for (int k = 0; k < 4; ++k) sum[j][k] = sum[j][k] < 0.f ? 0.f : sum[j][k];
      asm volatile("" : "+v"(sum[0]), "+v"(sum[1]), "+v"(sum[2]), "+v"(sum[3]));
      if ((unsigned)(now - tstart) < release) {
#pragma unroll
        for (int j = 0; j < 4; ++j) *(f4*)(buf + j * 256 + lane * 4) = sum[j];
        __builtin_amdgcn_fence(__ATOMIC_RELEASE, "workgroup");
        if (lane == 0) __hip_atomic_fetch_or(cnt + 5, 1u << red, __ATOMIC_RELAXED, __HIP_MEMORY_SCOPE_WORKGROUP);
        STAMPV(27, 1);
      } else {
        const char* o = sgpr_ptr((char*)obase + ((units >> (8 * red)) & 127) * (16 * SH * 4));
#pragma unroll
        for (int j = 0; j < 4; ++j) {
          if (LOCPLAIN && !remote) asm volatile("global_store_dwordx4 %0, %1, %2\n\ts_nop 1" :: "v"(vo + j * 4 * SH * 4), "v"(sum[j]), "s"(o) : "memory");
          else st16_sys_s(o, vo + j * 4 * SH * 4, __builtin_bit_cast(u4, sum[j]));
        }
        if (lane == 0) __hip_atomic_fetch_add(cnt + 4, 1u, __ATOMIC_RELAXED, __HIP_MEMORY_SCOPE_WORKGROUP);
        STAMPV(23 + red, 0);
      }
    }
  }
  STAMP(1);
  STAMP_FLUSH();
}

}

extern "C" void kernel_launch(int rank, int nranks, ncclComm_t comm, hipStream_t stream,
                              void* const* d_in, const int* in_sizes, int n_in,
                              void* d_out, int out_size,
                              const int* in_dtypes, int out_dtype,
                              void* const* d_out_all,
                              void* const* d_ws_all, size_t ws_bytes) {
  (void)comm; (void)in_dtypes; (void)out_dtype; (void)d_out;
  if (nranks != NR || (unsigned)rank >= (unsigned)NR || n_in != 2 || in_sizes[0] != SH * KK ||
      in_sizes[1] != KK * SH || out_size != MM * SH || ws_bytes < WS_END) {
    fprintf(stderr, "unexpected problem shape\n");
    return;
  }
  Params p;
  p.x = (const float*)d_in[0];
  p.w = (const float*)d_in[1];
  for (int i = 0; i < NR; ++i) { p.ws[i] = (char*)d_ws_all[i]; p.out[i] = (float*)d_out_all[i]; }
  p.myws = p.ws[rank];
  p.rank = rank;
  p.pad = 0;
  int ent[64], key[64], n = 0, m = 0;
  for (int g = 0; g < 16; ++g) p.otab[g] = 0;
  for (int s = 0; s < NR; ++s) {
    const bool same = ((s ^ rank) & 4) == 0;
    for (int u = 0; u < 16; ++u) {
      if (same != (u < NP)) { p.otab[m >> 2] |= (unsigned)(s * 16 + u) << (8 * (m & 3)); ++m; continue; }
      const int k = s == rank ? 0 : (same ? 10 * (u + 1) : 50 + 10 * (u - NP));
      int j = n++;
      for (; j > 0 && key[j - 1] > k; --j) { key[j] = key[j - 1]; ent[j] = ent[j - 1]; }
      key[j] = k; ent[j] = s * 16 + u;
    }
  }
  for (int g = 0; g < 16; ++g)
    p.tab[g] = (unsigned)ent[g] | (unsigned)ent[16 + g] << 8 | (unsigned)ent[32 + g] << 16 | (unsigned)ent[48 + g] << 24;
  hipLaunchKernelGGL(ag_gemm_kernel, dim3(NBLOCKS), dim3(NTHREADS), 0, stream, p);
  HIP_CHECK(hipGetLastError());
}
```

```cpp
#include <hip/hip_runtime.h>
#include <rccl/rccl.h>
#include <cstdio>
#include <cstdint>

#define HIP_CHECK(expr) do { hipError_t _e = (expr); if (_e != hipSuccess) { \
    fprintf(stderr, "HIP error %s at %s:%d\n", hipGetErrorString(_e), __FILE__, __LINE__); \
    return; } } while (0)

#define WPLP 8
#define WPLX 8
#define WIN 6
#define S0 6
#define LA 3
#define LOCDELAY 24
#define ZT0 300
#define ZMODE 0
#define ZWB 0
#define LOCPLAIN 0
#define ZSTEP 16
#define LDSTAG 4
#define OFULL 1550
#define REL0 1050
#define RELSTEP 28
#define RELROT 0
#define OPOLL0 1000
#define PROBE 4096
#define PRED 0
#define SLROT 1
#define GMAP 2
#define FRESH 1
#define PSLEEP 1
#define NP 11

namespace {

typedef _Float16 h8 __attribute__((ext_vector_type(8)));
typedef float f4 __attribute__((ext_vector_type(4)));
typedef unsigned u4 __attribute__((ext_vector_type(4)));
typedef unsigned long long u64;

constexpr int NR = 8;
constexpr int MM = 2048, KK = 2048;
constexpr int SH = 256;
constexpr int NTHREADS = 512, NBLOCKS = 256;
constexpr int NJ_P = NP * 8;
constexpr int NJ_X = 32 + (16 - NP) * 8;

constexpr int NS_P = 16, NS_X = 16;
static_assert(S0 >= 4 && S0 <= 8 && S0 * 8 <= NJ_P && S0 * 8 <= NJ_X && NJ_P <= 8 * NS_P && NJ_X <= 8 * NS_X && S0 < NS_X, "");

constexpr unsigned XU_B = 16 * KK * 2;
constexpr unsigned WS_B = 64 * KK * 2;
constexpr unsigned JOB_B = 8192;
constexpr unsigned XB_B = NR * 16 * XU_B;
constexpr unsigned WB_B = 8 * WS_B;
constexpr unsigned X_OFF = 0;
constexpr unsigned W_OFF = 3 * XB_B;
constexpr unsigned FL_STR = 128;
constexpr unsigned FW_OFF = W_OFF + 3 * WB_B;
constexpr unsigned MK_OFF = FW_OFF + 4 * 32 * FL_STR;
constexpr unsigned TK_OFF = MK_OFF + 4 * FL_STR;
constexpr unsigned TM_OFF = TK_OFF + 8 * FL_STR;
constexpr unsigned CT_OFF = TM_OFF + FL_STR;
constexpr unsigned ST_OFF = CT_OFF + NBLOCKS * 8 * 4;
constexpr unsigned WS_END = ST_OFF + NBLOCKS * 8 * 32 * 8;
#define DL 100000000ull
constexpr u64 DEADLINE = DL;
constexpr unsigned MARK = 0xBF800000u;

struct Params {
  const float* x;
  const float* w;
  char* myws;
  int rank;
  int pad;
  char* ws[NR];
  float* out[NR];
  unsigned tab[16];
  unsigned otab[16];
};

#define STAMPV(i, v) do {} while (0)
#define STAMP(i) do {} while (0)
#define STAMP_INIT() do {} while (0)
#define STAMP_FLUSH() do {} while (0)

template <int OFF>
__device__ __forceinline__ void st16_sys(char* a, u4 v) {
  asm volatile("global_store_dwordx4 %0, %1, off offset:%2 sc0 sc1\n\ts_nop 1"
               :: "v"(a), "v"(v), "i"(OFF) : "memory");
}
__device__ __forceinline__ const char* sgpr_ptr(const char* a) {
  const unsigned lo = __builtin_amdgcn_readfirstlane((unsigned)(u64)a), hi = __builtin_amdgcn_readfirstlane((unsigned)((u64)a >> 32));
  const char* r = (const char*)((u64)lo | (u64)hi << 32);
  asm volatile("s_nop 4" : "+s"(r));
  return r;
}
__device__ __forceinline__ void load16(f4 (&r)[16], const char* src, unsigned vo, unsigned sa, unsigned sb, unsigned sc) {
  const char* a = sgpr_ptr(src);
#pragma unroll
  for (int k = 0; k < 16; ++k)
    asm volatile("global_load_dwordx4 %0, %1, %2" : "=v"(r[k])
                 : "v"(vo + (k >> 3) * sa + ((k >> 1) & 3) * sb + (k & 1) * sc), "s"(a) : "memory");
}
__device__ __forceinline__ void st16_sys_s(const char* base, unsigned vo, u4 v) {
  asm volatile("global_store_dwordx4 %0, %1, %2 sc0 sc1\n\ts_nop 1" :: "v"(vo), "v"(v), "s"(base) : "memory");
}
__device__ __forceinline__ void st_flag(char* a, unsigned v) {
  asm volatile("s_mov_b64 exec, 1\n\tglobal_store_dword %0, %1, off sc0 sc1\n\ts_mov_b64 exec, -1"
               :: "v"(a), "v"(v) : "memory");
}
#define RAW_BARRIER() do { asm volatile("s_waitcnt lgkmcnt(0)" ::: "memory"); __builtin_amdgcn_s_barrier(); asm volatile("" ::: "memory"); } while (0)
#define CLEAR_VM() __builtin_amdgcn_s_waitcnt(0x0F70)
template <int N>
__device__ __forceinline__ void wait_vm() { asm volatile("s_waitcnt vmcnt(%0)" :: "i"(N) : "memory"); }
__device__ __forceinline__ unsigned next_call(unsigned* ctr, int lane) {
  const unsigned call = __builtin_amdgcn_readfirstlane(
                            __hip_atomic_load(ctr, __ATOMIC_RELAXED, __HIP_MEMORY_SCOPE_AGENT)) + 1u;
  if (lane == 0) __hip_atomic_store(ctr, call, __ATOMIC_RELAXED, __HIP_MEMORY_SCOPE_AGENT);
  return call;
}

struct Job {
  const char* src;
  char* dst;
  char* flag;
  bool isw;
};

__device__ __forceinline__ void job_load(f4 (&r)[16], const Job& jb, int lane) {
  const unsigned vo = jb.isw ? (lane >> 4) * (8 * SH * 4) + (lane & 15) * 16 : (lane & 15) * (KK * 4) + (lane >> 4) * 32;
  load16(r, jb.src, vo, jb.isw ? 32 * SH * 4 : 512, jb.isw ? 2 * SH * 4 : 128, jb.isw ? SH * 4 : 16);
}
__device__ __forceinline__ void pin16(f4 (&r)[16]) {
  asm volatile("" : "+v"(r[0]), "+v"(r[1]), "+v"(r[2]), "+v"(r[3]), "+v"(r[4]), "+v"(r[5]), "+v"(r[6]),
               "+v"(r[7]), "+v"(r[8]), "+v"(r[9]), "+v"(r[10]), "+v"(r[11]), "+v"(r[12]), "+v"(r[13]),
               "+v"(r[14]), "+v"(r[15]) :: "memory");
}
template <int F0, int F1>
__device__ __forceinline__ void job_cvt(u4 (&h)[8], const f4 (&r)[16], bool isw) {
  if (!isw) {
#pragma unroll
    for (int f = F0; f < F1; ++f) {
      h8 v;
#pragma unroll
      for (int j = 0; j < 8; ++j) v[j] = (_Float16)r[2 * f + (j >> 2)][j & 3];
      h[f] = __builtin_bit_cast(u4, v);
#pragma unroll
      for (int j = 0; j < 4; ++j) h[f][j] = h[f][j] ? h[f][j] : 0x80008000u;
    }
  } else {
#pragma unroll
    for (int f = F0; f < F1; ++f) {
      h8 v;
#pragma unroll
      for (int j = 0; j < 8; ++j) v[j] = (_Float16)r[(f >> 2) * 8 + j][f & 3];
      h[f] = __builtin_bit_cast(u4, v);
#pragma unroll
      for (int j = 0; j < 4; ++j) h[f][j] = h[f][j] ? h[f][j] : 0x80008000u;
    }
  }
}
__device__ __forceinline__ void job_store(const u4 (&h)[8], char* dst, int lane) {
  char* a = dst + lane * 16 + 4096;
  st16_sys<-4096>(a, h[0]); st16_sys<-3072>(a, h[1]); st16_sys<-2048>(a, h[2]); st16_sys<-1024>(a, h[3]);
  st16_sys<0>(a, h[4]);     st16_sys<1024>(a, h[5]);  st16_sys<2048>(a, h[6]);  st16_sys<3072>(a, h[7]);
}

__device__ __forceinline__ Job x_job(const Params& p, char* ws, int unit, int chunk, unsigned call) {
  Job j;
  j.src = (const char*)p.x + (size_t)unit * 16 * (KK * 4) + chunk * 256 * 4;
  j.dst = ws + X_OFF + (call % 3u) * XB_B + (unsigned)(p.rank * 16 + unit) * XU_B + chunk * JOB_B;
  j.flag = nullptr;
  j.isw = false;
  return j;
}
__device__ __forceinline__ Job w_job(const Params& p, char* ws, int slice, int slot, int pc, unsigned call) {
  Job j;
  j.src = (const char*)p.w + (size_t)pc * 64 * (SH * 4) + slice * 64 * 4;
  j.dst = ws + W_OFF + (call % 3u) * WB_B + (unsigned)slot * WS_B + pc * JOB_B;
  j.flag = ws + FW_OFF + ((slot & 3) * 32 + pc) * FL_STR;
  j.isw = true;
  return j;
}

__global__ void __launch_bounds__(NTHREADS, 2) ag_gemm_kernel(Params p) {
  __shared__ __attribute__((aligned(16))) float lds[4 * 8 * 1024 + 32];
  const int lane = threadIdx.x & 63;
  const int wave = __builtin_amdgcn_readfirstlane(threadIdx.x >> 6);
  const int b = blockIdx.x, xcd = b & 7, bi = b >> 3;
  const int rank = p.rank, grp = rank & 4, idx = rank & 3;
  char* const myws = p.myws;
  unsigned* const ctr = (unsigned*)(myws + CT_OFF) + b * 8 + wave;
  unsigned* const tmo = (unsigned*)(myws + TM_OFF);
  asm volatile("" :: "s"(p.x), "s"(p.w), "s"(myws), "s"(rank));
  const u64 tstart = __builtin_amdgcn_s_memrealtime();
  STAMP_INIT();
  STAMP(0);

  typedef __attribute__((address_space(3))) unsigned* lds_u32_p;
  typedef __attribute__((address_space(3))) u4* lds_u4_p;
  if (__builtin_expect(b < 56, 1)) {
    const int link = b >> 3, i = b & 7;
    const bool cross = link >= 3;
    const int dst = cross ? (grp ^ 4) + ((idx + link - 3) & 3) : grp + ((idx + 1 + link) & 3);
    const int nj = cross ? NJ_X : NJ_P, n = cross ? NS_X : NS_P;
    char* const ws = p.ws[dst & 7];
    auto mk = [&](int q, unsigned c) {
      return cross && q < 32 ? w_job(p, ws, (dst + SLROT * rank) & 3, 4 + idx, q, c)
                             : x_job(p, ws, (cross ? NP - 4 : 0) + (q >> 3), q & 7, c);
    };
    const lds_u32_p full = (lds_u32_p)(lds + 4 * 8192);
    unsigned call = 0;
    CLEAR_VM();
    if (__builtin_expect(wave < 2, 1)) {
      f4 r[16];
      u4 h[8];
      unsigned c;
      asm volatile("global_load_dword %0, %1, off sc1" : "=v"(c) : "v"(ctr) : "memory");
      job_load(r, mk(i + 8 * wave, 0u), lane);
      if (threadIdx.x < 32) full[threadIdx.x] = 0u;
      RAW_BARRIER();
      wait_vm<8>();
      asm volatile("" : "+v"(c), "+v"(r[0]), "+v"(r[1]), "+v"(r[2]), "+v"(r[3]), "+v"(r[4]), "+v"(r[5]),
                   "+v"(r[6]), "+v"(r[7]) :: "memory");
      call = __builtin_amdgcn_readfirstlane(c) + 1u;
      char* a = mk(i + 8 * wave, call).dst + lane * 16 + 4096;
      job_cvt<0, 4>(h, r, cross);
      st16_sys<-4096>(a, h[0]); st16_sys<-3072>(a, h[1]); st16_sys<-2048>(a, h[2]); st16_sys<-1024>(a, h[3]);
      wait_vm<4>();
      asm volatile("" : "+v"(r[8]), "+v"(r[9]), "+v"(r[10]), "+v"(r[11]), "+v"(r[12]), "+v"(r[13]),
                   "+v"(r[14]), "+v"(r[15]) :: "memory");
      job_cvt<4, 8>(h, r, cross);
      st16_sys<0>(a, h[4]); st16_sys<1024>(a, h[5]); st16_sys<2048>(a, h[6]); st16_sys<3072>(a, h[7]);
      st_flag((char*)ctr, call);
      STAMP(2);
    } else {
      RAW_BARRIER();
      if (LDSTAG) for (int k = 2; k < wave; ++k) __builtin_amdgcn_s_sleep(LDSTAG);
    }
    typedef __attribute__((address_space(3))) char* lds_p;
    typedef unsigned u2 __attribute__((ext_vector_type(2)));
    const lds_p ring = (lds_p)lds;
    const int sl0 = ((lane & 48) + ((lane & 15) ^ (lane >> 4))) * 16, sl1 = sl0 ^ 64;
    if (wave == 0) {
#pragma unroll 1
      for (int j = 2; j < n; ++j) {
        unsigned e;
        while ((e = __hip_atomic_load(full + j, __ATOMIC_ACQUIRE, __HIP_MEMORY_SCOPE_WORKGROUP)) == 0u)
          __builtin_amdgcn_s_sleep(1);
        const int q = (int)__builtin_amdgcn_readfirstlane(e) - 1;
        if (q >= nj) break;
        u4 h[8];
#pragma unroll
        for (int f = 0; f < 8; ++f) h[f] = *(lds_u4_p)(ring + j * JOB_B + f * 1024 + ((f & 1) ? sl1 : sl0));
        wait_vm<WIN>();
        job_store(h, mk(q, call).dst, lane);
        if (lane == 0) __hip_atomic_store(full + 25, (unsigned)j - 1u, __ATOMIC_RELAXED, __HIP_MEMORY_SCOPE_WORKGROUP);
      }
      if (lane == 0) __hip_atomic_store(full + 25, 1000u, __ATOMIC_RELAXED, __HIP_MEMORY_SCOPE_WORKGROUP);
      STAMP(28);
    } else {
#pragma unroll 1
      for (int j = wave == 1 ? S0 : wave < S0 ? wave : n; j < n; j = wave > 3 ? n : j < S0 ? S0 + wave - 1 : j + 3) {
        int q = i + 8 * j;
        if (j >= S0) {
          if (!call) call = next_call(ctr, lane);
          while ((int)__hip_atomic_load(full + 25, __ATOMIC_RELAXED, __HIP_MEMORY_SCOPE_WORKGROUP) < j - 2 - LA ||
                 (int)__hip_atomic_load(full + 26, __ATOMIC_RELAXED, __HIP_MEMORY_SCOPE_WORKGROUP) < j - S0)
            __builtin_amdgcn_s_sleep(2);
          if (__hip_atomic_load(full + 26, __ATOMIC_RELAXED, __HIP_MEMORY_SCOPE_WORKGROUP) >= 1000u) break;
          unsigned t = 0;
          if (lane == 0)
            t = __hip_atomic_fetch_add((unsigned*)(myws + TK_OFF + link * FL_STR), 1u, __ATOMIC_RELAXED,
                                       __HIP_MEMORY_SCOPE_AGENT);
          q = (int)(__builtin_amdgcn_readfirstlane(t) - (call - 1u) * (unsigned)(8 * (n - S0))) + 8 * S0;
          if (lane == 0)
            __hip_atomic_store(full + 26, q < nj ? (unsigned)(j + 1 - S0) : 1000u, __ATOMIC_RELAXED, __HIP_MEMORY_SCOPE_WORKGROUP);
          if (q >= nj) {
            if (lane == 0) {
              __hip_atomic_store(full + j, 1000u, __ATOMIC_RELAXED, __HIP_MEMORY_SCOPE_WORKGROUP);
              __hip_atomic_fetch_add((unsigned*)(myws + TK_OFF + link * FL_STR), (unsigned)(n - 1 - j), __ATOMIC_RELAXED,
                                     __HIP_MEMORY_SCOPE_AGENT);
            }
            break;
          }
        }
        const bool isw = cross && q < 32;
        f4 r[16];
        CLEAR_VM();
        load16(r, mk(q, 0u).src, isw ? (lane >> 4) * (8 * SH * 4) + (lane & 15) * 16 : lane * 16,
               isw ? 32 * SH * 4 : 8 * KK * 4, isw ? 2 * SH * 4 : 2 * KK * 4, isw ? SH * 4 : KK * 4);
        wait_vm<0>();
        pin16(r);
        const lds_p img = ring + j * JOB_B;
        if (isw) {
          u4 h[8];
          job_cvt<0, 8>(h, r, true);
#pragma unroll
          for (int f = 0; f < 8; ++f) *(lds_u4_p)(img + f * 1024 + ((f & 1) ? sl1 : sl0)) = h[f];
        } else {
          typedef _Float16 h4 __attribute__((ext_vector_type(4)));
          const int g = (lane >> 1) & 3, sw = ((lane >> 1) & 7);
          const lds_p a = img + (lane >> 3) * 1024 + g * 256 + (lane & 1) * 8;
#pragma unroll
          for (int k = 0; k < 16; ++k) {
            const h4 v = {(_Float16)r[k][0], (_Float16)r[k][1], (_Float16)r[k][2], (_Float16)r[k][3]};
            u2 w2 = __builtin_bit_cast(u2, v);
            w2[0] = w2[0] ? w2[0] : 0x80008000u;
            w2[1] = w2[1] ? w2[1] : 0x80008000u;
            *(__attribute__((address_space(3))) u2*)(a + (k ^ sw) * 16) = w2;
          }
        }
        if (cross && q == 31) {
          if (!call) call = next_call(ctr, lane);
          const unsigned* m = (const unsigned*)(myws + MK_OFF + (link - 3) * FL_STR);
          while (__hip_atomic_load(m, __ATOMIC_RELAXED, __HIP_MEMORY_SCOPE_AGENT) - (call - 1u) * 64u < 64u) {
            if (__builtin_amdgcn_s_memrealtime() - tstart > DEADLINE) {
              if (lane == 0) __hip_atomic_store(tmo, call, __ATOMIC_RELAXED, __HIP_MEMORY_SCOPE_AGENT);
              const u4 nan = {0x7E007E00u, 0x7E007E00u, 0x7E007E00u, 0x7E007E00u};
#pragma unroll
              for (int f = 0; f < 8; ++f) *(lds_u4_p)(img + f * 1024 + sl0) = nan;
              break;
            }
            __builtin_amdgcn_s_sleep(2);
          }
        }
        __builtin_amdgcn_fence(__ATOMIC_RELEASE, "workgroup");
        if (lane == 0) __hip_atomic_store(full + j, (unsigned)q + 1u, __ATOMIC_RELAXED, __HIP_MEMORY_SCOPE_WORKGROUP);
        STAMP(j < S0 ? 2 : 3);
      }
    }
    STAMP(1);
    STAMP_FLUSH();
    return;
  }
  if (bi < 16) {
    const int sid = b;
    if (sid >= 64) {
      const int hid = sid - 64;
      const unsigned call = next_call(ctr, lane);
      if (LOCDELAY) __builtin_amdgcn_s_sleep(LOCDELAY);
      if (wave < 4) {
        const int q = hid + 64 * (wave & 1);
        if (wave < 2 || q < NJ_P) {
          const Job jb = wave < 2 ? w_job(p, myws, q >> 5, q >> 5, q & 31, call) : x_job(p, myws, q >> 3, q & 7, call);
          f4 r[16];
          u4 h[8];
          CLEAR_VM();
          job_load(r, jb, lane);
          wait_vm<0>();
          pin16(r);
          job_cvt<0, 8>(h, r, jb.isw);
          job_store(h, jb.dst, lane);
          if (jb.isw) {
            wait_vm<0>();
            st_flag(jb.flag, call);
          }
        }
        STAMP(2);
        while (__builtin_amdgcn_s_memrealtime() - tstart < ZT0) __builtin_amdgcn_s_sleep(8);
        const int su = (p.tab[hid & 15] >> (8 * (hid >> 4))) & 127;
        char* a = myws + X_OFF + ((call + 2u) % 3u) * XB_B + su * XU_B + wave * (XU_B / 4) + lane * 16;
        const u4 z = {0u, 0u, 0u, 0u};
#pragma unroll 1
        for (int j = 0; j < 20; ++j) {
          if (j == 16) a = myws + W_OFF + ((call + 2u) % 3u) * WB_B + (4 + wave) * WS_B + hid * 4096 - 16384 + lane * 16;
          st16_sys<0>(a + j * 1024, z);
          if (ZSTEP) __builtin_amdgcn_s_sleep(ZSTEP);
        }
        if (ZWB && wave == 0 && hid < 8) {
          while (__builtin_amdgcn_s_memrealtime() - tstart < ZWB) __builtin_amdgcn_s_sleep(8);
          __builtin_amdgcn_fence(__ATOMIC_RELEASE, "agent");
          wait_vm<0>();
        }
      } else {
        const int col = ((idx + wave + SLROT * rank) & 3) * 64;
        const int lo = ((lane >> 4) * SH + col + (lane & 15) * 4) * 4;
        const int o0 = (int)((p.otab[hid >> 2] >> (8 * (hid & 3))) & 127) * (16 * SH * 4);
        char* const out = (char*)p.out[rank];
        const u4 mk = {MARK, MARK, MARK, MARK};
#pragma unroll
        for (int j = 0; j < 4; ++j) st16_sys<0>(out + o0 + lo + j * 4 * SH * 4, mk);
        wait_vm<0>();
        if (lane == 0)
          __hip_atomic_fetch_add((unsigned*)(myws + MK_OFF + (wave - 4) * FL_STR), 1u, __ATOMIC_RELAXED,
                                 __HIP_MEMORY_SCOPE_AGENT);
        STAMP(29);
        while (__builtin_amdgcn_s_memrealtime() - tstart < OPOLL0) __builtin_amdgcn_s_sleep(8);
        const __amdgpu_buffer_rsrc_t ro = __builtin_amdgcn_make_buffer_rsrc(out, 0, MM * SH * 4, 0x00020000);
        const unsigned* const one = lane == 0 ? (const unsigned*)(out + o0 + (15 * SH + col + 63) * 4) : tmo;
        bool full = false;
        for (;;) {
          bool left = true;
          if (full) {
            u4 v[4];
#pragma unroll
            for (int j = 0; j < 4; ++j) v[j] = __builtin_amdgcn_raw_buffer_load_b128(ro, lo + j * 4 * SH * 4, o0, 17);
            left = false;
#pragma unroll
            for (int j = 0; j < 4; ++j)
#pragma unroll
              for (int k = 0; k < 4; ++k) left |= v[j][k] == MARK;
          }
          const unsigned v1 = __hip_atomic_load(one, __ATOMIC_RELAXED, __HIP_MEMORY_SCOPE_SYSTEM);
          if (__ballot(left) == 0ull) break;
          const u64 el = __builtin_amdgcn_s_memrealtime() - tstart;
          if (__ballot(lane != 0 && v1 == call) != 0ull || el > DEADLINE) {
            if (lane == 0) {
              __hip_atomic_store(tmo, call, __ATOMIC_RELAXED, __HIP_MEMORY_SCOPE_AGENT);
              p.out[rank][(size_t)rank * SH * SH] = __builtin_nanf("");
            }
            break;
          }
          full = el > OFULL || __builtin_amdgcn_readfirstlane(v1) != MARK;
          asm volatile("" ::: "memory");
          if (!full) __builtin_amdgcn_s_sleep(4);
        }
      }
    }
    STAMP(1);
    STAMP_FLUSH();
    return;
  }

  const unsigned call = next_call(ctr, lane);
  const int ci = bi - 16;
  const int g = GMAP == 1 ? ci : GMAP == 2 ? (ci & 8) + xcd : xcd * 2 + (ci >> 3), slot = GMAP == 1 ? (xcd + ci) & 7 : ci & 7;
  const int odst = slot < 4 ? rank : (grp ^ 4) + (slot - 4);
  float* const obase = p.out[odst & 7] + (slot < 4 ? slot : (idx + SLROT * odst) & 3) * 64;
  const unsigned units = p.tab[g];
  const __amdgpu_buffer_rsrc_t rs = __builtin_amdgcn_make_buffer_rsrc(myws, 0, (int)FW_OFF, 0x00020000);
  const int voff = lane * 16;

  const lds_u32_p cnt = (lds_u32_p)(lds + 4 * 8192);
  if (threadIdx.x < 8) cnt[threadIdx.x] = 0u;
  __syncthreads();

  const bool remote = slot >= 4;
  h8 B[8][4];
  bool ok = true;
  {
    const int sb = W_OFF + (call % 3u) * WB_B + slot * WS_B + wave * 32768;
    const unsigned* a = (const unsigned*)(myws + (lane >= 4 ? TM_OFF : remote ? sb + lane * JOB_B + PROBE
                                                  : FW_OFF + (slot * 32 + wave * 4 + lane) * FL_STR));
    const unsigned least = remote ? 1u : call;
#pragma unroll 1
    for (;;) {
      const unsigned v = __hip_atomic_load(a, __ATOMIC_RELAXED, __HIP_MEMORY_SCOPE_SYSTEM);
      if (__ballot(lane >= 4 && v == call) != 0ull) ok = false;
      if (__builtin_amdgcn_s_memrealtime() - tstart > DEADLINE) {
        if (lane == 0) __hip_atomic_store(tmo, call, __ATOMIC_RELAXED, __HIP_MEMORY_SCOPE_AGENT);
        ok = false;
      }
      if (ok && __ballot(lane < 4 && v < least) != 0ull) {
        __builtin_amdgcn_s_sleep(PSLEEP);
        continue;
      }
      asm volatile("" ::: "memory");
      STAMP(2);
      unsigned lo = ~0u;
#pragma unroll
      for (int s = 0; s < 8; ++s)
#pragma unroll
        for (int e = 0; e < 4; ++e) {
          const u4 v4 = __builtin_amdgcn_raw_buffer_load_b128(rs, voff + e * 1024, sb + s * 4096, 17);
          B[s][e] = __builtin_bit_cast(h8, v4);
        }
      __builtin_amdgcn_sched_barrier(0);
#pragma unroll
      for (int s = 0; s < 8; ++s)
#pragma unroll
        for (int e = 0; e < 4; ++e) {
          const u4 v4 = __builtin_bit_cast(u4, B[s][e]);
          lo = min(min(lo, min(v4[0], v4[1])), min(v4[2], v4[3]));
        }
      if (!ok || __ballot(lo == 0u) == 0ull) break;
    }
    ok = __builtin_amdgcn_readfirstlane(ok) != 0;
    STAMP(19);
  }

  const unsigned xb = X_OFF + (call % 3u) * XB_B + wave * JOB_B;
  const unsigned lu = (units >> (8 * (lane & 3))) & 127;
  const bool early = lane == 3 && (wave >= PRED || ((lu & 15) != 0 && (lu & 15) != NP));
  const unsigned* const probe = (const unsigned*)(myws +
      (lane < 4 ? xb + lu * XU_B + PROBE - (early ? PRED * JOB_B : 0) : TM_OFF));
  const unsigned release = remote ? REL0 + RELSTEP * (15 - ((g + RELROT * slot) & 15)) : 0u;
  const unsigned vo = (voff & 0x300) * 4 + (voff & 0xF0);
  unsigned todo = 15u, rdy = 0u;
  u64 now = tstart;
#pragma unroll 1
  for (;;) {
    int red = -1;
    if (!rdy) {
      unsigned v = 0u;
      if (todo) v = __hip_atomic_load(probe, __ATOMIC_RELAXED, __HIP_MEMORY_SCOPE_SYSTEM);
      now = __builtin_amdgcn_s_memrealtime();
      const unsigned sent = __builtin_amdgcn_readfirstlane(__hip_atomic_load(cnt + 4, __ATOMIC_RELAXED, __HIP_MEMORY_SCOPE_WORKGROUP));
      const unsigned held = __builtin_amdgcn_readfirstlane(__hip_atomic_load(cnt + 5, __ATOMIC_RELAXED, __HIP_MEMORY_SCOPE_WORKGROUP));
      const unsigned tosum = __builtin_amdgcn_readfirstlane(__hip_atomic_load(cnt + 6, __ATOMIC_RELAXED, __HIP_MEMORY_SCOPE_WORKGROUP));
      rdy = (unsigned)__ballot(v != 0u) & todo;
      if (__ballot(lane >= 4 && v == call) != 0ull) { ok = false; rdy = todo; }
      if (now - tstart > DEADLINE) {
        if (lane == 0) __hip_atomic_store(tmo, call, __ATOMIC_RELAXED, __HIP_MEMORY_SCOPE_AGENT);
        if (!todo && !tosum && !held) break;
        ok = false;
        rdy = todo;
      }
      if (!rdy) {
        if (tosum) {
          const unsigned bit = tosum & (0u - tosum);
          unsigned old = 0u;
          if (lane == 0) old = __hip_atomic_fetch_and(cnt + 6, ~bit, __ATOMIC_ACQUIRE, __HIP_MEMORY_SCOPE_WORKGROUP);
          if (!(__builtin_amdgcn_readfirstlane(old) & bit)) continue;
          red = __builtin_ctz(bit);
        } else if (held && (unsigned)(now - tstart) >= release) {
          unsigned m = 0u;
          if (lane == 0) m = __hip_atomic_exchange(cnt + 5, 0u, __ATOMIC_ACQUIRE, __HIP_MEMORY_SCOPE_WORKGROUP);
          m = __builtin_amdgcn_readfirstlane(m);
          __builtin_amdgcn_fence(__ATOMIC_ACQUIRE, "workgroup");
#pragma unroll 1
          for (; m; m &= m - 1u) {
            const int t = __builtin_ctz(m);
            const float* buf = lds + t * 8192 + lane * 4;
            const char* o = sgpr_ptr((char*)obase + ((units >> (8 * t)) & 127) * (16 * SH * 4));
            f4 sum[4];
#pragma unroll
            for (int j = 0; j < 4; ++j) sum[j] = *(const f4*)(buf + j * 256);
#pragma unroll
            for (int j = 0; j < 4; ++j) st16_sys_s(o, vo + j * 4 * SH * 4, __builtin_bit_cast(u4, sum[j]));
            if (lane == 0) __hip_atomic_fetch_add(cnt + 4, 1u, __ATOMIC_RELAXED, __HIP_MEMORY_SCOPE_WORKGROUP);
            STAMPV(23 + t, 2);
          }
          continue;
        } else {
          if (!todo && sent == 4u) break;
          __builtin_amdgcn_s_sleep(PSLEEP);
          continue;
        }
      }
    }
    if (rdy) {
      const int t = __builtin_ctz(rdy);
      STAMPV(3 + 4 * it, t);
      const int sb = __builtin_amdgcn_readfirstlane(xb + ((units >> (8 * t)) & 127) * XU_B);
      asm volatile("" ::: "memory");
      u4 A[8];
#pragma unroll
      for (int s = 0; s < 8; ++s)
        A[s] = __builtin_amdgcn_raw_buffer_load_b128(rs, voff + (s & 3) * 1024, sb + (s >> 2) * 4096, 17);
      __builtin_amdgcn_sched_barrier(0);
      f4 acc[4];
#pragma unroll
      for (int e = 0; e < 4; ++e) acc[e] = f4{0.f, 0.f, 0.f, 0.f};
      unsigned lo = ~0u;
#pragma unroll
      for (int s = 0; s < 8; ++s) {
#pragma unroll
        for (int e = 0; e < 4; ++e)
          acc[e] = __builtin_amdgcn_mfma_f32_16x16x32_f16(__builtin_bit_cast(h8, A[s]), B[s][e], acc[e], 0, 0, 0);
        lo = min(min(lo, min(A[s][0], A[s][1])), min(A[s][2], A[s][3]));
      }
      const bool missing = __ballot(lo == 0u) != 0ull;
      if (ok & missing) {
        if (__builtin_amdgcn_s_memrealtime() - tstart <= DEADLINE) continue;
        if (lane == 0) __hip_atomic_store(tmo, call, __ATOMIC_RELAXED, __HIP_MEMORY_SCOPE_AGENT);
        ok = false;
      }
      rdy &= rdy - 1u;
      todo &= ~(1u << t);
      if (!ok) {
        const float nan = __builtin_nanf("");
#pragma unroll
        for (int e = 0; e < 4; ++e) acc[e] = f4{nan, nan, nan, nan};
      }
      float* slab = lds + t * 8192 + wave * 1024 + (lane >> 4) * 4 * 64 + (lane & 15) * 4;
#pragma unroll
      for (int r = 0; r < 4; ++r) *(f4*)(slab + r * 64) = f4{acc[0][r], acc[1][r], acc[2][r], acc[3][r]};
      unsigned old = 0;
      if (lane == 0) old = __hip_atomic_fetch_add(cnt + t, 1u, __ATOMIC_ACQ_REL, __HIP_MEMORY_SCOPE_WORKGROUP);
      STAMP(4 + 4 * it);
      if (__builtin_amdgcn_readfirstlane(old) == 7u) {
        if (todo) {
          if (lane == 0) __hip_atomic_fetch_or(cnt + 6, 1u << t, __ATOMIC_RELEASE, __HIP_MEMORY_SCOPE_WORKGROUP);
        } else {
          red = t;
        }
      }
    }
    if (red >= 0) {
      __builtin_amdgcn_fence(__ATOMIC_ACQUIRE, "workgroup");
      if (FRESH) now = __builtin_amdgcn_s_memrealtime();
      float* const buf = lds + red * 8192;
      f4 sum[4], pa[4][4];
#pragma unroll
      for (int hf = 0; hf < 2; ++hf) {
#pragma unroll
        for (int v = 0; v < 4; ++v)
#pragma unroll
          for (int j = 0; j < 4; ++j) pa[v][j] = *(const f4*)(buf + (hf * 4 + v) * 1024 + j * 256 + lane * 4);
        __builtin_amdgcn_sched_barrier(0);
#pragma unroll
        for (int j = 0; j < 4; ++j) {
          const f4 q = (pa[0][j] + pa[1][j]) + (pa[2][j] + pa[3][j]);
          sum[j] = hf ? sum[j] + q : q;
        }
        asm volatile("" : "+v"(sum[0]), "+v"(sum[1]), "+v"(sum[2]), "+v"(sum[3]));
      }
#pragma unroll
      for (int j = 0; j < 4; ++j)
#pragma unroll
        for (int k = 0; k < 4; ++k) sum[j][k] = sum[j][k] < 0.f ? 0.f : sum[j][k];
      asm volatile("" : "+v"(sum[0]), "+v"(sum[1]), "+v"(sum[2]), "+v"(sum[3]));
      if ((unsigned)(now - tstart) < release) {
#pragma unroll
        for (int j = 0; j < 4; ++j) *(f4*)(buf + j * 256 + lane * 4) = sum[j];
        __builtin_amdgcn_fence(__ATOMIC_RELEASE, "workgroup");
        if (lane == 0) __hip_atomic_fetch_or(cnt + 5, 1u << red, __ATOMIC_RELAXED, __HIP_MEMORY_SCOPE_WORKGROUP);
        STAMPV(27, 1);
      } else {
        const char* o = sgpr_ptr((char*)obase + ((units >> (8 * red)) & 127) * (16 * SH * 4));
#pragma unroll
        for (int j = 0; j < 4; ++j) {
          if (LOCPLAIN && !remote) asm volatile("global_store_dwordx4 %0, %1, %2\n\ts_nop 1" :: "v"(vo + j * 4 * SH * 4), "v"(sum[j]), "s"(o) : "memory");
          else st16_sys_s(o, vo + j * 4 * SH * 4, __builtin_bit_cast(u4, sum[j]));
        }
        if (lane == 0) __hip_atomic_fetch_add(cnt + 4, 1u, __ATOMIC_RELAXED, __HIP_MEMORY_SCOPE_WORKGROUP);
        STAMPV(23 + red, 0);
      }
    }
  }
  STAMP(1);
  STAMP_FLUSH();
}

}

extern "C" void kernel_launch(int rank, int nranks, ncclComm_t comm, hipStream_t stream,
                              void* const* d_in, const int* in_sizes, int n_in,
                              void* d_out, int out_size,
                              const int* in_dtypes, int out_dtype,
                              void* const* d_out_all,
                              void* const* d_ws_all, size_t ws_bytes) {
  (void)comm; (void)in_dtypes; (void)out_dtype; (void)d_out;
  if (nranks != NR || (unsigned)rank >= (unsigned)NR || n_in != 2 || in_sizes[0] != SH * KK ||
      in_sizes[1] != KK * SH || out_size != MM * SH || ws_bytes < WS_END) {
    fprintf(stderr, "unexpected problem shape\n");
    return;
  }
  Params p;
  p.x = (const float*)d_in[0];
  p.w = (const float*)d_in[1];
  for (int i = 0; i < NR; ++i) { p.ws[i] = (char*)d_ws_all[i]; p.out[i] = (float*)d_out_all[i]; }
  p.myws = p.ws[rank];
  p.rank = rank;
  p.pad = 0;
  int ent[64], key[64], n = 0, m = 0;
  for (int g = 0; g < 16; ++g) p.otab[g] = 0;
  for (int s = 0; s < NR; ++s) {
    const bool same = ((s ^ rank) & 4) == 0;
    for (int u = 0; u < 16; ++u) {
      if (same != (u < NP)) { p.otab[m >> 2] |= (unsigned)(s * 16 + u) << (8 * (m & 3)); ++m; continue; }
      const int k = s == rank ? 0 : (same ? 10 * (u + 1) : 50 + 10 * (u - NP));
      int j = n++;
      for (; j > 0 && key[j - 1] > k; --j) { key[j] = key[j - 1]; ent[j] = ent[j - 1]; }
      key[j] = k; ent[j] = s * 16 + u;
    }
  }
  for (int g = 0; g < 16; ++g)
    p.tab[g] = (unsigned)ent[g] | (unsigned)ent[16 + g] << 8 | (unsigned)ent[32 + g] << 16 | (unsigned)ent[48 + g] << 24;
  hipLaunchKernelGGL(ag_gemm_kernel, dim3(NBLOCKS), dim3(NTHREADS), 0, stream, p);
  HIP_CHECK(hipGetLastError());
}
```

```cpp
#include <hip/hip_runtime.h>
#include <rccl/rccl.h>
#include <cstdio>
#include <cstdint>

#define HIP_CHECK(expr) do { hipError_t _e = (expr); if (_e != hipSuccess) { \
    fprintf(stderr, "HIP error %s at %s:%d\n", hipGetErrorString(_e), __FILE__, __LINE__); \
    return; } } while (0)

#define WPLP 8
#define WPLX 8
#define WIN 6
#define S0 6
#define LA 3
#define LOCDELAY 24
#define ZT0 300
#define ZMODE 0
#define ZWB 0
#define LOCPLAIN 0
#define ZSTEP 16
#define LDSTAG 3
#define OFULL 1550
#define REL0 1060
#define RELSTEP 7
#define TSTEP 112
#define RELROT 0
#define OPOLL0 1000
#define PROBE 4096
#define PRED 0
#define SLROT 1
#define GMAP 2
#define FRESH 1
#define DMAPROBE 1
#define ISLEEP 4
#define PRIO 3
#define PSLEEP 2
#define NP 11

namespace {

typedef _Float16 h8 __attribute__((ext_vector_type(8)));
typedef float f4 __attribute__((ext_vector_type(4)));
typedef unsigned u4 __attribute__((ext_vector_type(4)));
typedef unsigned long long u64;

constexpr int NR = 8;
constexpr int MM = 2048, KK = 2048;
constexpr int SH = 256;
constexpr int NTHREADS = 512, NBLOCKS = 256;
constexpr int NJ_P = NP * 8;
constexpr int NJ_X = 32 + (16 - NP) * 8;

constexpr int NS_P = 16, NS_X = 16;
static_assert(S0 >= 4 && S0 <= 8 && S0 * 8 <= NJ_P && S0 * 8 <= NJ_X && NJ_P <= 8 * NS_P && NJ_X <= 8 * NS_X && S0 < NS_X, "");

constexpr unsigned XU_B = 16 * KK * 2;
constexpr unsigned WS_B = 64 * KK * 2;
constexpr unsigned JOB_B = 8192;
constexpr unsigned XB_B = NR * 16 * XU_B;
constexpr unsigned WB_B = 8 * WS_B;
constexpr unsigned X_OFF = 0;
constexpr unsigned W_OFF = 3 * XB_B;
constexpr unsigned FL_STR = 128;
constexpr unsigned FW_OFF = W_OFF + 3 * WB_B;
constexpr unsigned MK_OFF = FW_OFF + 4 * 32 * FL_STR;
constexpr unsigned TK_OFF = MK_OFF + 4 * FL_STR;
constexpr unsigned TM_OFF = TK_OFF + 8 * FL_STR;
constexpr unsigned CT_OFF = TM_OFF + FL_STR;
constexpr unsigned ST_OFF = CT_OFF + NBLOCKS * 8 * 4;
constexpr unsigned WS_END = ST_OFF + NBLOCKS * 8 * 32 * 8;
#define DL 100000000ull
constexpr u64 DEADLINE = DL;
constexpr unsigned MARK = 0xBF800000u;

struct Params {
  const float* x;
  const float* w;
  char* myws;
  int rank;
  int pad;
  char* ws[NR];
  float* out[NR];
  unsigned tab[16];
  unsigned otab[16];
};

#define STAMPV(i, v) do {} while (0)
#define STAMP(i) do {} while (0)
#define STAMP_INIT() do {} while (0)
#define STAMP_FLUSH() do {} while (0)
#define LT(i) do {} while (0)

template <int OFF>
__device__ __forceinline__ void st16_sys(char* a, u4 v) {
  asm volatile("global_store_dwordx4 %0, %1, off offset:%2 sc0 sc1\n\ts_nop 1"
               :: "v"(a), "v"(v), "i"(OFF) : "memory");
}
__device__ __forceinline__ const char* sgpr_ptr(const char* a) {
  const unsigned lo = __builtin_amdgcn_readfirstlane((unsigned)(u64)a), hi = __builtin_amdgcn_readfirstlane((unsigned)((u64)a >> 32));
  const char* r = (const char*)((u64)lo | (u64)hi << 32);
  asm volatile("s_nop 4" : "+s"(r));
  return r;
}
__device__ __forceinline__ void load16(f4 (&r)[16], const char* src, unsigned vo, unsigned sa, unsigned sb, unsigned sc) {
  const char* a = sgpr_ptr(src);
#pragma unroll
  for (int k = 0; k < 16; ++k)
    asm volatile("global_load_dwordx4 %0, %1, %2" : "=v"(r[k])
                 : "v"(vo + (k >> 3) * sa + ((k >> 1) & 3) * sb + (k & 1) * sc), "s"(a) : "memory");
}
__device__ __forceinline__ void st16_sys_s(const char* base, unsigned vo, u4 v) {
  asm volatile("global_store_dwordx4 %0, %1, %2 sc0 sc1\n\ts_nop 1" :: "v"(vo), "v"(v), "s"(base) : "memory");
}
__device__ __forceinline__ void st_flag(char* a, unsigned v) {
  asm volatile("s_mov_b64 exec, 1\n\tglobal_store_dword %0, %1, off sc0 sc1\n\ts_mov_b64 exec, -1"
               :: "v"(a), "v"(v) : "memory");
}
#define RAW_BARRIER() do { asm volatile("s_waitcnt lgkmcnt(0)" ::: "memory"); __builtin_amdgcn_s_barrier(); asm volatile("" ::: "memory"); } while (0)
#define CLEAR_VM() __builtin_amdgcn_s_waitcnt(0x0F70)
template <int N>
__device__ __forceinline__ void wait_vm() { asm volatile("s_waitcnt vmcnt(%0)" :: "i"(N) : "memory"); }
__device__ __forceinline__ unsigned next_call(unsigned* ctr, int lane) {
  const unsigned call = __builtin_amdgcn_readfirstlane(
                            __hip_atomic_load(ctr, __ATOMIC_RELAXED, __HIP_MEMORY_SCOPE_AGENT)) + 1u;
  if (lane == 0) __hip_atomic_store(ctr, call, __ATOMIC_RELAXED, __HIP_MEMORY_SCOPE_AGENT);
  return call;
}

struct Job {
  const char* src;
  char* dst;
  char* flag;
  bool isw;
};

__device__ __forceinline__ void job_load(f4 (&r)[16], const Job& jb, int lane) {
  const unsigned vo = jb.isw ? (lane >> 4) * (8 * SH * 4) + (lane & 15) * 16 : (lane & 15) * (KK * 4) + (lane >> 4) * 32;
  load16(r, jb.src, vo, jb.isw ? 32 * SH * 4 : 512, jb.isw ? 2 * SH * 4 : 128, jb.isw ? SH * 4 : 16);
}
__device__ __forceinline__ void pin16(f4 (&r)[16]) {
  asm volatile("" : "+v"(r[0]), "+v"(r[1]), "+v"(r[2]), "+v"(r[3]), "+v"(r[4]), "+v"(r[5]), "+v"(r[6]),
               "+v"(r[7]), "+v"(r[8]), "+v"(r[9]), "+v"(r[10]), "+v"(r[11]), "+v"(r[12]), "+v"(r[13]),
               "+v"(r[14]), "+v"(r[15]) :: "memory");
}
template <int F0, int F1>
__device__ __forceinline__ void job_cvt(u4 (&h)[8], const f4 (&r)[16], bool isw) {
  if (!isw) {
#pragma unroll
    for (int f = F0; f < F1; ++f) {
      h8 v;
#pragma unroll
      for (int j = 0; j < 8; ++j) v[j] = (_Float16)r[2 * f + (j >> 2)][j & 3];
      h[f] = __builtin_bit_cast(u4, v);
#pragma unroll
      for (int j = 0; j < 4; ++j) h[f][j] = h[f][j] ? h[f][j] : 0x80008000u;
    }
  } else {
#pragma unroll
    for (int f = F0; f < F1; ++f) {
      h8 v;
#pragma unroll
      for (int j = 0; j < 8; ++j) v[j] = (_Float16)r[(f >> 2) * 8 + j][f & 3];
      h[f] = __builtin_bit_cast(u4, v);
#pragma unroll
      for (int j = 0; j < 4; ++j) h[f][j] = h[f][j] ? h[f][j] : 0x80008000u;
    }
  }
}
__device__ __forceinline__ void job_store(const u4 (&h)[8], char* dst, int lane) {
  char* a = dst + lane * 16 + 4096;
  st16_sys<-4096>(a, h[0]); st16_sys<-3072>(a, h[1]); st16_sys<-2048>(a, h[2]); st16_sys<-1024>(a, h[3]);
  st16_sys<0>(a, h[4]);     st16_sys<1024>(a, h[5]);  st16_sys<2048>(a, h[6]);  st16_sys<3072>(a, h[7]);
}

__device__ __forceinline__ Job x_job(const Params& p, char* ws, int unit, int chunk, unsigned call) {
  Job j;
  j.src = (const char*)p.x + (size_t)unit * 16 * (KK * 4) + chunk * 256 * 4;
  j.dst = ws + X_OFF + (call % 3u) * XB_B + (unsigned)(p.rank * 16 + unit) * XU_B + chunk * JOB_B;
  j.flag = nullptr;
  j.isw = false;
  return j;
}
__device__ __forceinline__ Job w_job(const Params& p, char* ws, int slice, int slot, int pc, unsigned call) {
  Job j;
  j.src = (const char*)p.w + (size_t)pc * 64 * (SH * 4) + slice * 64 * 4;
  j.dst = ws + W_OFF + (call % 3u) * WB_B + (unsigned)slot * WS_B + pc * JOB_B;
  j.flag = ws + FW_OFF + ((slot & 3) * 32 + pc) * FL_STR;
  j.isw = true;
  return j;
}

__global__ void __launch_bounds__(NTHREADS, 2) ag_gemm_kernel(Params p) {
  __shared__ __attribute__((aligned(16))) float lds[4 * 8 * 1024 + 80];
  const int lane = threadIdx.x & 63;
  const int wave = __builtin_amdgcn_readfirstlane(threadIdx.x >> 6);
  const int b = blockIdx.x, xcd = b & 7, bi = b >> 3;
  const int rank = p.rank, grp = rank & 4, idx = rank & 3;
  char* const myws = p.myws;
  unsigned* const ctr = (unsigned*)(myws + CT_OFF) + b * 8 + wave;
  unsigned* const tmo = (unsigned*)(myws + TM_OFF);
  asm volatile("" :: "s"(p.x), "s"(p.w), "s"(myws), "s"(rank));
  const u64 tstart = __builtin_amdgcn_s_memrealtime();
  STAMP_INIT();
  STAMP(0);

  typedef __attribute__((address_space(3))) unsigned* lds_u32_p;
  typedef __attribute__((address_space(3))) u4* lds_u4_p;
  if (__builtin_expect(b < 56, 1)) {
    const int link = b >> 3, i = b & 7;
    const bool cross = link >= 3;
    const int dst = cross ? (grp ^ 4) + ((idx + link - 3) & 3) : grp + ((idx + 1 + link) & 3);
    const int nj = cross ? NJ_X : NJ_P, n = cross ? NS_X : NS_P;
    char* const ws = p.ws[dst & 7];
    auto mk = [&](int q, unsigned c) {
      return cross && q < 32 ? w_job(p, ws, (dst + SLROT * rank) & 3, 4 + idx, q, c)
                             : x_job(p, ws, (cross ? NP - 4 : 0) + (q >> 3), q & 7, c);
    };
    const lds_u32_p full = (lds_u32_p)(lds + 4 * 8192);
    unsigned call = 0;
    CLEAR_VM();
    if (__builtin_expect(wave < 2, 1)) {
      f4 r[16];
      u4 h[8];
      unsigned c;
      asm volatile("global_load_dword %0, %1, off sc1" : "=v"(c) : "v"(ctr) : "memory");
      job_load(r, mk(i + 8 * wave, 0u), lane);
      if (threadIdx.x < 32) full[threadIdx.x] = 0u;
      RAW_BARRIER();
      wait_vm<8>();
      asm volatile("" : "+v"(c), "+v"(r[0]), "+v"(r[1]), "+v"(r[2]), "+v"(r[3]), "+v"(r[4]), "+v"(r[5]),
                   "+v"(r[6]), "+v"(r[7]) :: "memory");
      call = __builtin_amdgcn_readfirstlane(c) + 1u;
      char* a = mk(i + 8 * wave, call).dst + lane * 16 + 4096;
      job_cvt<0, 4>(h, r, cross);
      st16_sys<-4096>(a, h[0]); st16_sys<-3072>(a, h[1]); st16_sys<-2048>(a, h[2]); st16_sys<-1024>(a, h[3]);
      wait_vm<4>();
      asm volatile("" : "+v"(r[8]), "+v"(r[9]), "+v"(r[10]), "+v"(r[11]), "+v"(r[12]), "+v"(r[13]),
                   "+v"(r[14]), "+v"(r[15]) :: "memory");
      job_cvt<4, 8>(h, r, cross);
      st16_sys<0>(a, h[4]); st16_sys<1024>(a, h[5]); st16_sys<2048>(a, h[6]); st16_sys<3072>(a, h[7]);
      st_flag((char*)ctr, call);
      LT(0);
      STAMP(2);
    } else {
      RAW_BARRIER();
      if (LDSTAG) for (int k = 2; k < wave; ++k) __builtin_amdgcn_s_sleep(LDSTAG);
    }
    typedef __attribute__((address_space(3))) char* lds_p;
    typedef unsigned u2 __attribute__((ext_vector_type(2)));
    const lds_p ring = (lds_p)lds;
    const int sl0 = ((lane & 48) + ((lane & 15) ^ (lane >> 4))) * 16, sl1 = sl0 ^ 64;
    if (wave == 0) {
#pragma unroll 1
      for (int j = 2; j < n; ++j) {
        unsigned e;
        while ((e = __hip_atomic_load(full + j, __ATOMIC_ACQUIRE, __HIP_MEMORY_SCOPE_WORKGROUP)) == 0u)
          __builtin_amdgcn_s_sleep(1);
        const int q = (int)__builtin_amdgcn_readfirstlane(e) - 1;
        if (q >= nj) break;
        u4 h[8];
#pragma unroll
        for (int f = 0; f < 8; ++f) h[f] = *(lds_u4_p)(ring + j * JOB_B + f * 1024 + ((f & 1) ? sl1 : sl0));
        wait_vm<WIN>();
        job_store(h, mk(q, call).dst, lane);
        if (lane == 0) __hip_atomic_store(full + 25, (unsigned)j - 1u, __ATOMIC_RELAXED, __HIP_MEMORY_SCOPE_WORKGROUP);
      }
      if (lane == 0) __hip_atomic_store(full + 25, 1000u, __ATOMIC_RELAXED, __HIP_MEMORY_SCOPE_WORKGROUP);
      STAMP(28);
    } else {
#pragma unroll 1
      for (int j = wave == 1 ? S0 : wave < S0 ? wave : n; j < n; j = wave > 3 ? n : j < S0 ? S0 + wave - 1 : j + 3) {
        int q = i + 8 * j;
        if (j >= S0) {
          if (!call) call = next_call(ctr, lane);
          while ((int)__hip_atomic_load(full + 25, __ATOMIC_RELAXED, __HIP_MEMORY_SCOPE_WORKGROUP) < j - 2 - LA ||
                 (int)__hip_atomic_load(full + 26, __ATOMIC_RELAXED, __HIP_MEMORY_SCOPE_WORKGROUP) < j - S0)
            __builtin_amdgcn_s_sleep(2);
          if (__hip_atomic_load(full + 26, __ATOMIC_RELAXED, __HIP_MEMORY_SCOPE_WORKGROUP) >= 1000u) break;
          unsigned t = 0;
          if (lane == 0)
            t = __hip_atomic_fetch_add((unsigned*)(myws + TK_OFF + link * FL_STR), 1u, __ATOMIC_RELAXED,
                                       __HIP_MEMORY_SCOPE_AGENT);
          q = (int)(__builtin_amdgcn_readfirstlane(t) - (call - 1u) * (unsigned)(8 * (n - S0))) + 8 * S0;
          if (lane == 0)
            __hip_atomic_store(full + 26, q < nj ? (unsigned)(j + 1 - S0) : 1000u, __ATOMIC_RELAXED, __HIP_MEMORY_SCOPE_WORKGROUP);
          if (q >= nj) {
            if (lane == 0) {
              __hip_atomic_store(full + j, 1000u, __ATOMIC_RELAXED, __HIP_MEMORY_SCOPE_WORKGROUP);
              __hip_atomic_fetch_add((unsigned*)(myws + TK_OFF + link * FL_STR), (unsigned)(n - 1 - j), __ATOMIC_RELAXED,
                                     __HIP_MEMORY_SCOPE_AGENT);
            }
            break;
          }
        }
        const bool isw = cross && q < 32;
        f4 r[16];
        CLEAR_VM();
        load16(r, mk(q, 0u).src, isw ? (lane >> 4) * (8 * SH * 4) + (lane & 15) * 16 : lane * 16,
               isw ? 32 * SH * 4 : 8 * KK * 4, isw ? 2 * SH * 4 : 2 * KK * 4, isw ? SH * 4 : KK * 4);
        wait_vm<0>();
        pin16(r);
        const lds_p img = ring + j * JOB_B;
        if (isw) {
          u4 h[8];
          job_cvt<0, 8>(h, r, true);
#pragma unroll
          for (int f = 0; f < 8; ++f) *(lds_u4_p)(img + f * 1024 + ((f & 1) ? sl1 : sl0)) = h[f];
        } else {
          typedef _Float16 h4 __attribute__((ext_vector_type(4)));
          const int g = (lane >> 1) & 3, sw = ((lane >> 1) & 7);
          const lds_p a = img + (lane >> 3) * 1024 + g * 256 + (lane & 1) * 8;
#pragma unroll
          for (int k = 0; k < 16; ++k) {
            const h4 v = {(_Float16)r[k][0], (_Float16)r[k][1], (_Float16)r[k][2], (_Float16)r[k][3]};
            u2 w2 = __builtin_bit_cast(u2, v);
            w2[0] = w2[0] ? w2[0] : 0x80008000u;
            w2[1] = w2[1] ? w2[1] : 0x80008000u;
            *(__attribute__((address_space(3))) u2*)(a + (k ^ sw) * 16) = w2;
          }
        }
        if (cross && q == 31) {
          if (!call) call = next_call(ctr, lane);
          const unsigned* m = (const unsigned*)(myws + MK_OFF + (link - 3) * FL_STR);
          while (__hip_atomic_load(m, __ATOMIC_RELAXED, __HIP_MEMORY_SCOPE_AGENT) - (call - 1u) * 64u < 64u) {
            if (__builtin_amdgcn_s_memrealtime() - tstart > DEADLINE) {
              if (lane == 0) __hip_atomic_store(tmo, call, __ATOMIC_RELAXED, __HIP_MEMORY_SCOPE_AGENT);
              const u4 nan = {0x7E007E00u, 0x7E007E00u, 0x7E007E00u, 0x7E007E00u};
#pragma unroll
              for (int f = 0; f < 8; ++f) *(lds_u4_p)(img + f * 1024 + sl0) = nan;
              break;
            }
            __builtin_amdgcn_s_sleep(2);
          }
        }
        __builtin_amdgcn_fence(__ATOMIC_RELEASE, "workgroup");
        if (lane == 0) __hip_atomic_store(full + j, (unsigned)q + 1u, __ATOMIC_RELAXED, __HIP_MEMORY_SCOPE_WORKGROUP);
        STAMP(j < S0 ? 2 : 3);
      }
    }
    STAMP(1);
    STAMP_FLUSH();
    return;
  }
  if (bi < 16) {
    const int sid = b;
    if (sid >= 64) {
      const int hid = sid - 64;
      const unsigned call = next_call(ctr, lane);
      if (LOCDELAY) __builtin_amdgcn_s_sleep(LOCDELAY);
      if (wave < 4) {
        const int q = hid + 64 * (wave & 1);
        if (wave < 2 || q < NJ_P) {
          const Job jb = wave < 2 ? w_job(p, myws, q >> 5, q >> 5, q & 31, call) : x_job(p, myws, q >> 3, q & 7, call);
          f4 r[16];
          u4 h[8];
          CLEAR_VM();
          job_load(r, jb, lane);
          wait_vm<0>();
          pin16(r);
          job_cvt<0, 8>(h, r, jb.isw);
          job_store(h, jb.dst, lane);
          if (jb.isw) {
            wait_vm<0>();
            st_flag(jb.flag, call);
          }
        }
        STAMP(2);
        while (__builtin_amdgcn_s_memrealtime() - tstart < ZT0) __builtin_amdgcn_s_sleep(8);
        const int su = (p.tab[hid & 15] >> (8 * (hid >> 4))) & 127;
        char* a = myws + X_OFF + ((call + 2u) % 3u) * XB_B + su * XU_B + wave * (XU_B / 4) + lane * 16;
        const u4 z = {0u, 0u, 0u, 0u};
#pragma unroll 1
        for (int j = 0; j < 20; ++j) {
          if (j == 16) a = myws + W_OFF + ((call + 2u) % 3u) * WB_B + (4 + wave) * WS_B + hid * 4096 - 16384 + lane * 16;
          st16_sys<0>(a + j * 1024, z);
          if (ZSTEP) __builtin_amdgcn_s_sleep(ZSTEP);
        }
        if (ZWB && wave == 0 && hid < 8) {
          while (__builtin_amdgcn_s_memrealtime() - tstart < ZWB) __builtin_amdgcn_s_sleep(8);
          __builtin_amdgcn_fence(__ATOMIC_RELEASE, "agent");
          wait_vm<0>();
        }
      } else {
        const int col = ((idx + wave + SLROT * rank) & 3) * 64;
        const int lo = ((lane >> 4) * SH + col + (lane & 15) * 4) * 4;
        const int o0 = (int)((p.otab[hid >> 2] >> (8 * (hid & 3))) & 127) * (16 * SH * 4);
        char* const out = (char*)p.out[rank];
        const u4 mk = {MARK, MARK, MARK, MARK};
#pragma unroll
        for (int j = 0; j < 4; ++j) st16_sys<0>(out + o0 + lo + j * 4 * SH * 4, mk);
        wait_vm<0>();
        if (lane == 0)
          __hip_atomic_fetch_add((unsigned*)(myws + MK_OFF + (wave - 4) * FL_STR), 1u, __ATOMIC_RELAXED,
                                 __HIP_MEMORY_SCOPE_AGENT);
        STAMP(29);
        while (__builtin_amdgcn_s_memrealtime() - tstart < OPOLL0) __builtin_amdgcn_s_sleep(8);
        const __amdgpu_buffer_rsrc_t ro = __builtin_amdgcn_make_buffer_rsrc(out, 0, MM * SH * 4, 0x00020000);
        const unsigned* const one = lane == 0 ? (const unsigned*)(out + o0 + (15 * SH + col + 63) * 4) : tmo;
        auto give_up = [&](unsigned v1, u64 el) {
          if (__ballot(lane != 0 && v1 == call) == 0ull && el <= DEADLINE) return false;
          if (lane == 0) {
            __hip_atomic_store(tmo, call, __ATOMIC_RELAXED, __HIP_MEMORY_SCOPE_AGENT);
            p.out[rank][(size_t)rank * SH * SH] = __builtin_nanf("");
          }
          return true;
        };
        bool dead = false;
        for (;;) {
          const unsigned v1 = __hip_atomic_load(one, __ATOMIC_RELAXED, __HIP_MEMORY_SCOPE_SYSTEM);
          const u64 el = __builtin_amdgcn_s_memrealtime() - tstart;
          if ((dead = give_up(v1, el))) break;
          if (el > OFULL || __builtin_amdgcn_readfirstlane(v1) != MARK) break;
          __builtin_amdgcn_s_sleep(4);
        }
        auto fetch = [&](u4 (&v)[4]) {
          asm volatile("" ::: "memory");
#pragma unroll
          for (int j = 0; j < 4; ++j) v[j] = __builtin_amdgcn_raw_buffer_load_b128(ro, lo + j * 4 * SH * 4, o0, 17);
        };
        auto complete = [&](const u4 (&v)[4]) {
          bool left = false;
#pragma unroll
          for (int j = 0; j < 4; ++j)
#pragma unroll
            for (int k = 0; k < 4; ++k) left |= v[j][k] == MARK;
          return __ballot(left) == 0ull;
        };
        if (!dead) {
          u4 va[4];
          fetch(va);
#pragma unroll 1
          for (int k = 1; !complete(va); ++k) {
            if ((k & 7) == 0 && give_up(__hip_atomic_load(one, __ATOMIC_RELAXED, __HIP_MEMORY_SCOPE_SYSTEM),
                                        __builtin_amdgcn_s_memrealtime() - tstart)) break;
            fetch(va);
          }
        }
      }
    }
    STAMP(1);
    STAMP_FLUSH();
    return;
  }

  const unsigned call = next_call(ctr, lane);
  const int ci = bi - 16;
  const int g = GMAP == 1 ? ci : GMAP == 2 ? (ci & 8) + xcd : xcd * 2 + (ci >> 3), slot = GMAP == 1 ? (xcd + ci) & 7 : ci & 7;
  const int odst = slot < 4 ? rank : (grp ^ 4) + (slot - 4);
  float* const obase = p.out[odst & 7] + (slot < 4 ? slot : (idx + SLROT * odst) & 3) * 64;
  const unsigned units = p.tab[g];
  const __amdgpu_buffer_rsrc_t rs = __builtin_amdgcn_make_buffer_rsrc(myws, 0, (int)FW_OFF, 0x00020000);
  const int voff = lane * 16;

  const lds_u32_p cnt = (lds_u32_p)(lds + 4 * 8192);
  if (threadIdx.x < 80) cnt[threadIdx.x] = 0u;
  __syncthreads();

  const bool remote = slot >= 4;
  h8 B[8][4];
  bool ok = true;
  {
    const int sb = W_OFF + (call % 3u) * WB_B + slot * WS_B + wave * 32768;
    const unsigned* a = (const unsigned*)(myws + (lane >= 4 ? TM_OFF : remote ? sb + lane * JOB_B + PROBE
                                                  : FW_OFF + (slot * 32 + wave * 4 + lane) * FL_STR));
    const unsigned least = remote ? 1u : call;
#pragma unroll 1
    for (;;) {
      const unsigned v = __hip_atomic_load(a, __ATOMIC_RELAXED, __HIP_MEMORY_SCOPE_SYSTEM);
      if (__ballot(lane >= 4 && v == call) != 0ull) ok = false;
      if (__builtin_amdgcn_s_memrealtime() - tstart > DEADLINE) {
        if (lane == 0) __hip_atomic_store(tmo, call, __ATOMIC_RELAXED, __HIP_MEMORY_SCOPE_AGENT);
        ok = false;
      }
      if (ok && __ballot(lane < 4 && v < least) != 0ull) {
        __builtin_amdgcn_s_sleep(1);
        continue;
      }
      asm volatile("" ::: "memory");
      STAMP(2);
      unsigned lo = ~0u;
#pragma unroll
      for (int s = 0; s < 8; ++s)
#pragma unroll
        for (int e = 0; e < 4; ++e) {
          const u4 v4 = __builtin_amdgcn_raw_buffer_load_b128(rs, voff + e * 1024, sb + s * 4096, 17);
          B[s][e] = __builtin_bit_cast(h8, v4);
        }
      __builtin_amdgcn_sched_barrier(0);
#pragma unroll
      for (int s = 0; s < 8; ++s)
#pragma unroll
        for (int e = 0; e < 4; ++e) {
          const u4 v4 = __builtin_bit_cast(u4, B[s][e]);
          lo = min(min(lo, min(v4[0], v4[1])), min(v4[2], v4[3]));
        }
      if (!ok || __ballot(lo == 0u) == 0ull) break;
    }
    ok = __builtin_amdgcn_readfirstlane(ok) != 0;
    STAMP(19);
  }

  const unsigned xb = X_OFF + (call % 3u) * XB_B + wave * JOB_B;
  const unsigned lu = (units >> (8 * (lane & 3))) & 127;
  const bool early = lane == 3 && (wave >= PRED || ((lu & 15) != 0 && (lu & 15) != NP));
  const unsigned* const probe = (const unsigned*)(myws +
      (lane < 4 ? xb + lu * XU_B + PROBE - (early ? PRED * JOB_B : 0) : TM_OFF));
  const unsigned release = remote ? REL0 + RELSTEP * (15 - ((g + RELROT * slot) & 15)) : 0u;
  const unsigned vo = (voff & 0x300) * 4 + (voff & 0xF0);
  const unsigned pw = (unsigned)(size_t)(cnt + 16 + 8 * wave);
  const lds_u32_p look = cnt + (lane < 5 ? 16 + 8 * wave + lane : lane < 8 ? lane - 1 : 4);
  unsigned todo = 15u, rdy = 0u;
  u64 now = tstart;
#pragma unroll 1
  for (;;) {
    int red = -1;
    if (!rdy) {
      if (PRIO) __builtin_amdgcn_s_setprio(0);
      if (todo) {
        unsigned keep;
        asm volatile("s_mov_b32 %0, m0\n\ts_mov_b32 m0, %2\n\ts_mov_b64 exec, %3\n\t"
                     "global_load_lds_dword %1, off sc0 sc1\n\ts_mov_b64 exec, -1\n\ts_mov_b32 m0, %0"
                     : "=&s"(keep) : "v"(probe), "s"(pw), "s"((u64)(todo | 16u)) : "memory");
        __builtin_amdgcn_s_sleep(PSLEEP);
      }
      now = __builtin_amdgcn_s_memrealtime();
      const unsigned lv = __hip_atomic_load(look, __ATOMIC_RELAXED, __HIP_MEMORY_SCOPE_WORKGROUP);
      const unsigned v = lv;
      const unsigned sent = __builtin_amdgcn_readlane(lv, 5), held = __builtin_amdgcn_readlane(lv, 6);
      const unsigned tosum = __builtin_amdgcn_readlane(lv, 7);
      rdy = (unsigned)__ballot(v != 0u) & todo;
      if (__builtin_amdgcn_readlane(v, 4) == call) { ok = false; rdy = todo; }
      if (now - tstart > DEADLINE) {
        if (lane == 0) __hip_atomic_store(tmo, call, __ATOMIC_RELAXED, __HIP_MEMORY_SCOPE_AGENT);
        if (!todo && !tosum && !held) break;
        ok = false;
        rdy = todo;
      }
      if (!rdy) {
        if (tosum) {
          const unsigned bit = tosum & (0u - tosum);
          unsigned old = 0u;
          if (lane == 0) old = __hip_atomic_fetch_and(cnt + 6, ~bit, __ATOMIC_ACQUIRE, __HIP_MEMORY_SCOPE_WORKGROUP);
          if (!(__builtin_amdgcn_readfirstlane(old) & bit)) continue;
          red = __builtin_ctz(bit);
        } else if (held && (unsigned)(now - tstart) >= release + TSTEP * sent) {
          unsigned m = 0u;
          if (lane == 0) {
            if (TSTEP) {
              unsigned exp = sent;
              if (__hip_atomic_compare_exchange_strong(cnt + 4, &exp, sent + 1u, __ATOMIC_RELAXED, __ATOMIC_RELAXED, __HIP_MEMORY_SCOPE_WORKGROUP)) {
                const unsigned bit = held & (0u - held);
                m = __hip_atomic_fetch_and(cnt + 5, ~bit, __ATOMIC_ACQUIRE, __HIP_MEMORY_SCOPE_WORKGROUP) & bit;
                if (!m) __hip_atomic_fetch_sub(cnt + 4, 1u, __ATOMIC_RELAXED, __HIP_MEMORY_SCOPE_WORKGROUP);
              }
            } else {
              m = __hip_atomic_exchange(cnt + 5, 0u, __ATOMIC_ACQUIRE, __HIP_MEMORY_SCOPE_WORKGROUP);
            }
          }
          m = __builtin_amdgcn_readfirstlane(m);
          __builtin_amdgcn_fence(__ATOMIC_ACQUIRE, "workgroup");
#pragma unroll 1
          for (; m; m &= m - 1u) {
            const int t = __builtin_ctz(m);
            const float* buf = lds + t * 8192 + lane * 4;
            const char* o = sgpr_ptr((char*)obase + ((units >> (8 * t)) & 127) * (16 * SH * 4));
            f4 sum[4];
#pragma unroll
            for (int j = 0; j < 4; ++j) sum[j] = *(const f4*)(buf + j * 256);
#pragma unroll
            for (int j = 0; j < 4; ++j) st16_sys_s(o, vo + j * 4 * SH * 4, __builtin_bit_cast(u4, sum[j]));
            if (!TSTEP && lane == 0) __hip_atomic_fetch_add(cnt + 4, 1u, __ATOMIC_RELAXED, __HIP_MEMORY_SCOPE_WORKGROUP);
            STAMPV(23 + t, 2);
            LT(3);
          }
          continue;
        } else {
          if (!todo && sent == 4u) break;
          if (!todo) __builtin_amdgcn_s_sleep(ISLEEP);
          else if (!DMAPROBE) __builtin_amdgcn_s_sleep(1);
          continue;
        }
      }
    }
    if (rdy) {
      const int t = __builtin_ctz(rdy);
      if (PRIO) __builtin_amdgcn_s_setprio(PRIO);
      STAMPV(3 + 4 * it, t);
      LT(0);
      const int sb = __builtin_amdgcn_readfirstlane(xb + ((units >> (8 * t)) & 127) * XU_B);
      asm volatile("" ::: "memory");
      u4 A[8];
#pragma unroll
      for (int s = 0; s < 8; ++s)
        A[s] = __builtin_amdgcn_raw_buffer_load_b128(rs, voff + (s & 3) * 1024, sb + (s >> 2) * 4096, 17);
      __builtin_amdgcn_sched_barrier(0);
      f4 acc[4];
#pragma unroll
      for (int e = 0; e < 4; ++e) acc[e] = f4{0.f, 0.f, 0.f, 0.f};
      unsigned lo = ~0u;
#pragma unroll
      for (int s = 0; s < 8; ++s) {
#pragma unroll
        for (int e = 0; e < 4; ++e)
          acc[e] = __builtin_amdgcn_mfma_f32_16x16x32_f16(__builtin_bit_cast(h8, A[s]), B[s][e], acc[e], 0, 0, 0);
        lo = min(min(lo, min(A[s][0], A[s][1])), min(A[s][2], A[s][3]));
      }
      const bool missing = __ballot(lo == 0u) != 0ull;
      if (ok & missing) {
        if (__builtin_amdgcn_s_memrealtime() - tstart <= DEADLINE) continue;
        if (lane == 0) __hip_atomic_store(tmo, call, __ATOMIC_RELAXED, __HIP_MEMORY_SCOPE_AGENT);
        ok = false;
      }
      rdy &= rdy - 1u;
      todo &= ~(1u << t);
      if (!ok) {
        const float nan = __builtin_nanf("");
#pragma unroll
        for (int e = 0; e < 4; ++e) acc[e] = f4{nan, nan, nan, nan};
      }
      float* slab = lds + t * 8192 + wave * 1024 + (lane >> 4) * 4 * 64 + (lane & 15) * 4;
#pragma unroll
      for (int r = 0; r < 4; ++r) *(f4*)(slab + r * 64) = f4{acc[0][r], acc[1][r], acc[2][r], acc[3][r]};
      unsigned old = 0;
      if (lane == 0) old = __hip_atomic_fetch_add(cnt + t, 1u, __ATOMIC_ACQ_REL, __HIP_MEMORY_SCOPE_WORKGROUP);
      STAMP(4 + 4 * it);
      if (__builtin_amdgcn_readfirstlane(old) == 7u) {
        if (todo) {
          if (lane == 0) __hip_atomic_fetch_or(cnt + 6, 1u << t, __ATOMIC_RELEASE, __HIP_MEMORY_SCOPE_WORKGROUP);
        } else {
          red = t;
        }
      }
    }
    if (red >= 0) {
      LT(1);
      __builtin_amdgcn_fence(__ATOMIC_ACQUIRE, "workgroup");
      if (FRESH) now = __builtin_amdgcn_s_memrealtime();
      const unsigned nsent = TSTEP ? __builtin_amdgcn_readfirstlane(__hip_atomic_load(cnt + 4, __ATOMIC_RELAXED, __HIP_MEMORY_SCOPE_WORKGROUP)) : 0u;
      float* const buf = lds + red * 8192;
      f4 sum[4], pa[4][4];
#pragma unroll
      for (int hf = 0; hf < 2; ++hf) {
#pragma unroll
        for (int v = 0; v < 4; ++v)
#pragma unroll
          for (int j = 0; j < 4; ++j) pa[v][j] = *(const f4*)(buf + (hf * 4 + v) * 1024 + j * 256 + lane * 4);
        __builtin_amdgcn_sched_barrier(0);
#pragma unroll
        for (int j = 0; j < 4; ++j) {
          const f4 q = (pa[0][j] + pa[1][j]) + (pa[2][j] + pa[3][j]);
          sum[j] = hf ? sum[j] + q : q;
        }
        asm volatile("" : "+v"(sum[0]), "+v"(sum[1]), "+v"(sum[2]), "+v"(sum[3]));
      }
#pragma unroll
      for (int j = 0; j < 4; ++j) sum[j] = __builtin_elementwise_maximum(sum[j], f4{0.f, 0.f, 0.f, 0.f});
      asm volatile("" : "+v"(sum[0]), "+v"(sum[1]), "+v"(sum[2]), "+v"(sum[3]));
      if ((unsigned)(now - tstart) < release + (remote ? TSTEP * nsent : 0u)) {
#pragma unroll
        for (int j = 0; j < 4; ++j) *(f4*)(buf + j * 256 + lane * 4) = sum[j];
        __builtin_amdgcn_fence(__ATOMIC_RELEASE, "workgroup");
        if (lane == 0) __hip_atomic_fetch_or(cnt + 5, 1u << red, __ATOMIC_RELAXED, __HIP_MEMORY_SCOPE_WORKGROUP);
        STAMPV(27, 1);
      } else {
        const char* o = sgpr_ptr((char*)obase + ((units >> (8 * red)) & 127) * (16 * SH * 4));
#pragma unroll
        for (int j = 0; j < 4; ++j) {
          if (LOCPLAIN && !remote) asm volatile("global_store_dwordx4 %0, %1, %2\n\ts_nop 1" :: "v"(vo + j * 4 * SH * 4), "v"(sum[j]), "s"(o) : "memory");
          else st16_sys_s(o, vo + j * 4 * SH * 4, __builtin_bit_cast(u4, sum[j]));
        }
        if (lane == 0) __hip_atomic_fetch_add(cnt + 4, 1u, __ATOMIC_RELAXED, __HIP_MEMORY_SCOPE_WORKGROUP);
        STAMPV(23 + red, 0);
        LT(2);
      }
    }
  }
  STAMP(1);
  STAMP_FLUSH();
}

}

extern "C" void kernel_launch(int rank, int nranks, ncclComm_t comm, hipStream_t stream,
                              void* const* d_in, const int* in_sizes, int n_in,
                              void* d_out, int out_size,
                              const int* in_dtypes, int out_dtype,
                              void* const* d_out_all,
                              void* const* d_ws_all, size_t ws_bytes) {
  (void)comm; (void)in_dtypes; (void)out_dtype; (void)d_out;
  if (nranks != NR || (unsigned)rank >= (unsigned)NR || n_in != 2 || in_sizes[0] != SH * KK ||
      in_sizes[1] != KK * SH || out_size != MM * SH || ws_bytes < WS_END) {
    fprintf(stderr, "unexpected problem shape\n");
    return;
  }
  Params p;
  p.x = (const float*)d_in[0];
  p.w = (const float*)d_in[1];
  for (int i = 0; i < NR; ++i) { p.ws[i] = (char*)d_ws_all[i]; p.out[i] = (float*)d_out_all[i]; }
  p.myws = p.ws[rank];
  p.rank = rank;
  p.pad = 0;
  int ent[64], key[64], n = 0, m = 0;
  for (int g = 0; g < 16; ++g) p.otab[g] = 0;
  for (int s = 0; s < NR; ++s) {
    const bool same = ((s ^ rank) & 4) == 0;
    for (int u = 0; u < 16; ++u) {
      if (same != (u < NP)) { p.otab[m >> 2] |= (unsigned)(s * 16 + u) << (8 * (m & 3)); ++m; continue; }
      const int k = s == rank ? 0 : (same ? 10 * (u + 1) : 50 + 10 * (u - NP));
      int j = n++;
      for (; j > 0 && key[j - 1] > k; --j) { key[j] = key[j - 1]; ent[j] = ent[j - 1]; }
      key[j] = k; ent[j] = s * 16 + u;
    }
  }
  for (int g = 0; g < 16; ++g)
    p.tab[g] = (unsigned)ent[g] | (unsigned)ent[16 + g] << 8 | (unsigned)ent[32 + g] << 16 | (unsigned)ent[48 + g] << 24;
  hipLaunchKernelGGL(ag_gemm_kernel, dim3(NBLOCKS), dim3(NTHREADS), 0, stream, p);
  HIP_CHECK(hipGetLastError());
}
```

```cpp
#include <hip/hip_runtime.h>
#include <rccl/rccl.h>
#include <cstdio>
#include <cstdint>

#define HIP_CHECK(expr) do { hipError_t _e = (expr); if (_e != hipSuccess) { \
    fprintf(stderr, "HIP error %s at %s:%d\n", hipGetErrorString(_e), __FILE__, __LINE__); \
    return; } } while (0)

constexpr int NP = 11;
constexpr int PW = 6;
constexpr int S0 = 6;
constexpr int LA = 3;
constexpr int LDSTAG = 1;
constexpr int TLOCAL = 100;
constexpr int TCLEAR = 300;
constexpr int ZSTEP = 16;
constexpr int REL0 = 1045;
constexpr int RELSTEP = 8;
constexpr int TSTEP = 126;
constexpr int OPOLL0 = 1000;
constexpr int OFULL = 1550;

namespace {

typedef _Float16 h8 __attribute__((ext_vector_type(8)));
typedef float f4 __attribute__((ext_vector_type(4)));
typedef unsigned u4 __attribute__((ext_vector_type(4)));
typedef unsigned long long u64;

constexpr int NR = 8;
constexpr int MM = 2048, KK = 2048;
constexpr int SH = 256;
constexpr int NTHREADS = 512, NBLOCKS = 256;
constexpr int NJ_P = NP * 8;
constexpr int NJ_X = 32 + (16 - NP) * 8;

constexpr int NS_P = 16, NS_X = 16;
static_assert(S0 >= 4 && S0 <= 8 && S0 * 8 <= NJ_P && S0 * 8 <= NJ_X && NJ_P <= 8 * NS_P && NJ_X <= 8 * NS_X && S0 < NS_X, "");

constexpr unsigned XU_B = 16 * KK * 2;
constexpr unsigned WS_B = 64 * KK * 2;
constexpr unsigned JOB_B = 8192;
constexpr unsigned PROBE = 4096;
constexpr unsigned XB_B = NR * 16 * XU_B;
constexpr unsigned WB_B = 8 * WS_B;
constexpr unsigned X_OFF = 0;
constexpr unsigned W_OFF = 3 * XB_B;
constexpr unsigned EB_B = 16384;
constexpr unsigned E_OFF = W_OFF + 3 * WB_B;
constexpr unsigned FL_STR = 128;
constexpr unsigned FW_OFF = E_OFF + 3 * EB_B;
constexpr unsigned MK_OFF = FW_OFF + 4 * 32 * FL_STR;
constexpr unsigned TK_OFF = MK_OFF + 4 * FL_STR;
constexpr unsigned TM_OFF = TK_OFF + 8 * FL_STR;
constexpr unsigned CT_OFF = TM_OFF + FL_STR;
constexpr unsigned WS_END = CT_OFF + NBLOCKS * 8 * 4;
constexpr u64 DEADLINE = 100000000ull;
constexpr unsigned MARK = 0xBF800000u;

struct Params {
  const float* x;
  const float* w;
  char* myws;
  int rank;
  int pad;
  char* ws[NR];
  float* out[NR];
  unsigned tab[16];
  unsigned otab[16];
};

template <int OFF>
__device__ __forceinline__ void st16_sys(char* a, u4 v) {
  asm volatile("global_store_dwordx4 %0, %1, off offset:%2 sc0 sc1\n\ts_nop 1"
               :: "v"(a), "v"(v), "i"(OFF) : "memory");
}
__device__ __forceinline__ const char* sgpr_ptr(const char* a) {
  const unsigned lo = __builtin_amdgcn_readfirstlane((unsigned)(u64)a), hi = __builtin_amdgcn_readfirstlane((unsigned)((u64)a >> 32));
  const char* r = (const char*)((u64)lo | (u64)hi << 32);
  asm volatile("s_nop 4" : "+s"(r));
  return r;
}
__device__ __forceinline__ void load16(f4 (&r)[16], const char* src, unsigned vo, unsigned sa, unsigned sb, unsigned sc) {
  const char* a = sgpr_ptr(src);
#pragma unroll
  for (int k = 0; k < 16; ++k)
    asm volatile("global_load_dwordx4 %0, %1, %2" : "=v"(r[k])
                 : "v"(vo + (k >> 3) * sa + ((k >> 1) & 3) * sb + (k & 1) * sc), "s"(a) : "memory");
}
__device__ __forceinline__ void st16_sys_s(const char* base, unsigned vo, u4 v) {
  asm volatile("global_store_dwordx4 %0, %1, %2 sc0 sc1\n\ts_nop 1" :: "v"(vo), "v"(v), "s"(base) : "memory");
}
__device__ __forceinline__ void st_flag(char* a, unsigned v) {
  asm volatile("s_mov_b64 exec, 1\n\tglobal_store_dword %0, %1, off sc0 sc1\n\ts_mov_b64 exec, -1"
               :: "v"(a), "v"(v) : "memory");
}
#define RAW_BARRIER() do { asm volatile("s_waitcnt lgkmcnt(0)" ::: "memory"); __builtin_amdgcn_s_barrier(); asm volatile("" ::: "memory"); } while (0)
#define CLEAR_VM() __builtin_amdgcn_s_waitcnt(0x0F70)
template <int N>
__device__ __forceinline__ void wait_vm() { asm volatile("s_waitcnt vmcnt(%0)" :: "i"(N) : "memory"); }
__device__ __forceinline__ unsigned next_call(unsigned* ctr, int lane) {
  const unsigned call = __builtin_amdgcn_readfirstlane(
                            __hip_atomic_load(ctr, __ATOMIC_RELAXED, __HIP_MEMORY_SCOPE_AGENT)) + 1u;
  if (lane == 0) __hip_atomic_store(ctr, call, __ATOMIC_RELAXED, __HIP_MEMORY_SCOPE_AGENT);
  return call;
}

struct Job {
  const char* src;
  char* dst;
  char* flag;
  char* exps;
  bool isw;
};

__device__ __forceinline__ void job_load(f4 (&r)[16], const Job& jb, int lane) {
  const unsigned vo = jb.isw ? (lane >> 4) * (8 * SH * 4) + (lane & 15) * 16 : (lane & 15) * (KK * 4) + (lane >> 4) * 32;
  load16(r, jb.src, vo, jb.isw ? 32 * SH * 4 : 512, jb.isw ? 2 * SH * 4 : 128, jb.isw ? SH * 4 : 16);
}
__device__ __forceinline__ void pin16(f4 (&r)[16]) {
  asm volatile("" : "+v"(r[0]), "+v"(r[1]), "+v"(r[2]), "+v"(r[3]), "+v"(r[4]), "+v"(r[5]), "+v"(r[6]),
               "+v"(r[7]), "+v"(r[8]), "+v"(r[9]), "+v"(r[10]), "+v"(r[11]), "+v"(r[12]), "+v"(r[13]),
               "+v"(r[14]), "+v"(r[15]) :: "memory");
}
template <int K0, int K1, int W, class PUT>
__device__ __forceinline__ void fit_range(f4 (&r)[16], PUT put) {
  float m = 0.f;
#pragma unroll
  for (int k = K0; k < K1; k += 2) {
    const float a = r[k][0], b = r[k][1], c = r[k][2], d = r[k][3];
    const float e = r[k + 1][0], f = r[k + 1][1], g = r[k + 1][2], h = r[k + 1][3];
    asm("v_max3_f32 %0, |%1|, |%2|, %0\n\tv_max3_f32 %0, |%3|, |%4|, %0\n\t"
        "v_max3_f32 %0, |%5|, |%6|, %0\n\tv_max3_f32 %0, |%7|, |%8|, %0"
        : "+v"(m) : "v"(a), "v"(b), "v"(c), "v"(d), "v"(e), "v"(f), "v"(g), "v"(h));
  }
  const u64 big = __ballot(m > 65504.f), some = __ballot(m >= 0x1p-9f);
  if (__builtin_expect(!big && (W == 64 ? some != 0ull : (unsigned)some && (some >> 32)), 1)) return;
#pragma unroll 1
  for (int d = 1; d < W; d <<= 1) m = fmaxf(m, __shfl_xor(m, d));
  const int ef = __builtin_bit_cast(int, m) >> 23;
  const int e = m == 0.f || ef == 255 || (m >= 0x1p-9f && m <= 65504.f) ? 0 : ef - (127 + 4);
#pragma unroll
  for (int k = K0; k < K1; ++k)
#pragma unroll
    for (int j = 0; j < 4; ++j) r[k][j] = __builtin_ldexpf(r[k][j], -e);
  put(e);
}
__device__ __forceinline__ void put_exp(char* a, int e) {
  if (e != 0) {
    st_flag(a, (unsigned)e);
    wait_vm<0>();
  }
}
template <int F0, int F1>
__device__ __forceinline__ void job_cvt(u4 (&h)[8], const f4 (&r)[16], bool isw) {
  if (!isw) {
#pragma unroll
    for (int f = F0; f < F1; ++f) {
      h8 v;
#pragma unroll
      for (int j = 0; j < 8; ++j) v[j] = (_Float16)r[2 * f + (j >> 2)][j & 3];
      h[f] = __builtin_bit_cast(u4, v);
    }
  } else {
#pragma unroll
    for (int f = F0; f < F1; ++f) {
      h8 v;
#pragma unroll
      for (int j = 0; j < 8; ++j) v[j] = (_Float16)r[(f >> 2) * 8 + j][f & 3];
      h[f] = __builtin_bit_cast(u4, v);
    }
  }
  unsigned lo = ~0u;
#pragma unroll
  for (int f = F0; f < F1; ++f) lo = min(min(lo, min(h[f][0], h[f][1])), min(h[f][2], h[f][3]));
  if (__builtin_expect(__ballot(lo == 0u) != 0ull, 0)) {
#pragma unroll
    for (int f = F0; f < F1; ++f)
#pragma unroll
      for (int j = 0; j < 4; ++j) h[f][j] = h[f][j] ? h[f][j] : 0x80008000u;
  }
}
__device__ __forceinline__ void job_store(const u4 (&h)[8], char* dst, int lane) {
  char* a = dst + lane * 16 + 4096;
  st16_sys<-4096>(a, h[0]); st16_sys<-3072>(a, h[1]); st16_sys<-2048>(a, h[2]); st16_sys<-1024>(a, h[3]);
  st16_sys<0>(a, h[4]);     st16_sys<1024>(a, h[5]);  st16_sys<2048>(a, h[6]);  st16_sys<3072>(a, h[7]);
}

__device__ __forceinline__ Job x_job(const Params& p, char* ws, int unit, int chunk, unsigned call) {
  Job j;
  j.src = (const char*)p.x + (size_t)unit * 16 * (KK * 4) + chunk * 256 * 4;
  j.dst = ws + X_OFF + (call % 3u) * XB_B + (unsigned)(p.rank * 16 + unit) * XU_B + chunk * JOB_B;
  j.flag = nullptr;
  j.exps = ws + E_OFF + (call % 3u) * EB_B + ((p.rank * 16 + unit) * 8 + chunk) * 8;
  j.isw = false;
  return j;
}
__device__ __forceinline__ Job w_job(const Params& p, char* ws, int slice, int slot, int pc, unsigned call) {
  Job j;
  j.src = (const char*)p.w + (size_t)pc * 64 * (SH * 4) + slice * 64 * 4;
  j.dst = ws + W_OFF + (call % 3u) * WB_B + (unsigned)slot * WS_B + pc * JOB_B;
  j.flag = ws + FW_OFF + ((slot & 3) * 32 + pc) * FL_STR;
  j.exps = ws + E_OFF + (call % 3u) * EB_B + 8192 + (slot * 32 + pc) * 8;
  j.isw = true;
  return j;
}

__global__ void __launch_bounds__(NTHREADS, 2) ag_gemm_kernel(Params p) {
  __shared__ __attribute__((aligned(16))) float lds[4 * 8 * 1024 + 80];
  const int lane = threadIdx.x & 63;
  const int wave = __builtin_amdgcn_readfirstlane(threadIdx.x >> 6);
  const int b = blockIdx.x, xcd = b & 7, bi = b >> 3;
  const int rank = p.rank, grp = rank & 4, idx = rank & 3;
  char* const myws = p.myws;
  unsigned* const ctr = (unsigned*)(myws + CT_OFF) + b * 8 + wave;
  unsigned* const tmo = (unsigned*)(myws + TM_OFF);
  asm volatile("" :: "s"(p.x), "s"(p.w), "s"(myws), "s"(rank));
  const u64 tstart = __builtin_amdgcn_s_memrealtime();

  typedef __attribute__((address_space(3))) unsigned* lds_u32_p;
  typedef __attribute__((address_space(3))) u4* lds_u4_p;
  if (__builtin_expect(b < 56, 1)) {
    const int link = b >> 3, i = b & 7;
    const bool cross = link >= 3;
    const int dst = cross ? (grp ^ 4) + ((idx + link - 3) & 3) : grp + ((idx + 1 + link) & 3);
    const int nj = cross ? NJ_X : NJ_P, n = cross ? NS_X : NS_P;
    char* const ws = p.ws[dst & 7];
    auto mk = [&](int q, unsigned c) {
      return cross && q < 32 ? w_job(p, ws, (dst + rank) & 3, 4 + idx, q, c)
                             : x_job(p, ws, (cross ? NP - 4 : 0) + (q >> 3), q & 7, c);
    };
    const lds_u32_p full = (lds_u32_p)(lds + 4 * 8192);
    unsigned call = 0;
    CLEAR_VM();
    if (__builtin_expect(wave == 0, 1)) {
      f4 r[16];
      u4 h[8];
      unsigned c;
      asm volatile("global_load_dword %0, %1, off sc1" : "=v"(c) : "v"(ctr) : "memory");
      job_load(r, mk(i, 0u), lane);
      if (threadIdx.x < 32) full[threadIdx.x] = 0u;
      RAW_BARRIER();
      wait_vm<8>();
      asm volatile("" : "+v"(c), "+v"(r[0]), "+v"(r[1]), "+v"(r[2]), "+v"(r[3]), "+v"(r[4]), "+v"(r[5]),
                   "+v"(r[6]), "+v"(r[7]) :: "memory");
      call = __builtin_amdgcn_readfirstlane(c) + 1u;
      char* a = mk(i, call).dst + lane * 16 + 4096;
      fit_range<0, 8, 64>(r, [&](int e) { put_exp(mk(i, call).exps, __builtin_amdgcn_readfirstlane(e)); });
      job_cvt<0, 4>(h, r, cross);
      st16_sys<-4096>(a, h[0]); st16_sys<-3072>(a, h[1]); st16_sys<-2048>(a, h[2]); st16_sys<-1024>(a, h[3]);
      wait_vm<4>();
      asm volatile("" : "+v"(r[8]), "+v"(r[9]), "+v"(r[10]), "+v"(r[11]), "+v"(r[12]), "+v"(r[13]),
                   "+v"(r[14]), "+v"(r[15]) :: "memory");
      fit_range<8, 16, 64>(r, [&](int e) { put_exp(mk(i, call).exps + 4, __builtin_amdgcn_readfirstlane(e)); });
      job_cvt<4, 8>(h, r, cross);
      st16_sys<0>(a, h[4]); st16_sys<1024>(a, h[5]); st16_sys<2048>(a, h[6]); st16_sys<3072>(a, h[7]);
      st_flag((char*)ctr, call);
    } else if (wave >= 3) {
      RAW_BARRIER();
      for (int k = 3; k < wave; ++k) __builtin_amdgcn_s_sleep(LDSTAG);
    }
    typedef __attribute__((address_space(3))) char* lds_p;
    typedef unsigned u2 __attribute__((ext_vector_type(2)));
    const lds_p ring = (lds_p)lds;
    const int sl0 = ((lane & 48) + ((lane & 15) ^ (lane >> 4))) * 16, sl1 = sl0 ^ 64;
    if (wave == 0) {
#pragma unroll 1
      for (int j = 1; j < n; ++j) {
        unsigned e;
        while ((e = __hip_atomic_load(full + j, __ATOMIC_ACQUIRE, __HIP_MEMORY_SCOPE_WORKGROUP)) == 0u)
          __builtin_amdgcn_s_sleep(1);
        const int q = (int)__builtin_amdgcn_readfirstlane(e) - 1;
        if (q >= nj) break;
        u4 h[8];
#pragma unroll
        for (int f = 0; f < 8; ++f) h[f] = *(lds_u4_p)(ring + j * JOB_B + f * 1024 + ((f & 1) ? sl1 : sl0));
        char* a = mk(q, call).dst + lane * 16 + 4096;
        wait_vm<PW>(); st16_sys<-4096>(a, h[0]); wait_vm<PW>(); st16_sys<-3072>(a, h[1]);
        wait_vm<PW>(); st16_sys<-2048>(a, h[2]); wait_vm<PW>(); st16_sys<-1024>(a, h[3]);
        wait_vm<PW>(); st16_sys<0>(a, h[4]);     wait_vm<PW>(); st16_sys<1024>(a, h[5]);
        wait_vm<PW>(); st16_sys<2048>(a, h[6]);  wait_vm<PW>(); st16_sys<3072>(a, h[7]);
        if (lane == 0) __hip_atomic_store(full + 25, (unsigned)j + 1u, __ATOMIC_RELAXED, __HIP_MEMORY_SCOPE_WORKGROUP);
      }
      if (lane == 0) __hip_atomic_store(full + 25, 1000u, __ATOMIC_RELAXED, __HIP_MEMORY_SCOPE_WORKGROUP);
    } else {
#pragma unroll 1
      for (int j = wave < S0 ? wave : n; j < n; j = wave > 3 ? n : j < S0 ? S0 + wave - 1 : j + 3) {
        int q = i + 8 * j;
        if (j >= S0) {
          if (!call) call = next_call(ctr, lane);
          while ((int)__hip_atomic_load(full + 25, __ATOMIC_RELAXED, __HIP_MEMORY_SCOPE_WORKGROUP) < j - LA ||
                 (int)__hip_atomic_load(full + 26, __ATOMIC_RELAXED, __HIP_MEMORY_SCOPE_WORKGROUP) < j - S0)
            __builtin_amdgcn_s_sleep(2);
          if (__hip_atomic_load(full + 26, __ATOMIC_RELAXED, __HIP_MEMORY_SCOPE_WORKGROUP) >= 1000u) break;
          unsigned t = 0;
          if (lane == 0)
            t = __hip_atomic_fetch_add((unsigned*)(myws + TK_OFF + link * FL_STR), 1u, __ATOMIC_RELAXED,
                                       __HIP_MEMORY_SCOPE_AGENT);
          q = (int)(__builtin_amdgcn_readfirstlane(t) - (call - 1u) * (unsigned)(8 * (n - S0))) + 8 * S0;
          if (q < 8 * S0) q = nj;
          if (lane == 0)
            __hip_atomic_store(full + 26, q < nj ? (unsigned)(j + 1 - S0) : 1000u, __ATOMIC_RELAXED, __HIP_MEMORY_SCOPE_WORKGROUP);
          if (q >= nj) {
            if (lane == 0) {
              __hip_atomic_store(full + j, 1000u, __ATOMIC_RELAXED, __HIP_MEMORY_SCOPE_WORKGROUP);
              __hip_atomic_fetch_add((unsigned*)(myws + TK_OFF + link * FL_STR), (unsigned)(n - 1 - j), __ATOMIC_RELAXED,
                                     __HIP_MEMORY_SCOPE_AGENT);
            }
            break;
          }
        }
        const bool isw = cross && q < 32;
        f4 r[16];
        CLEAR_VM();
        load16(r, mk(q, 0u).src, isw ? (lane >> 4) * (8 * SH * 4) + (lane & 15) * 16 : lane * 16,
               isw ? 32 * SH * 4 : 8 * KK * 4, isw ? 2 * SH * 4 : 2 * KK * 4, isw ? SH * 4 : KK * 4);
        if (j < 3) RAW_BARRIER();
        wait_vm<0>();
        pin16(r);
        const lds_p img = ring + j * JOB_B;
        auto put = [&](int half, int e) {
          if (!call) call = next_call(ctr, lane);
          put_exp(mk(q, call).exps + 4 * half, e);
        };
        if (isw) {
          u4 h[8];
          fit_range<0, 8, 64>(r, [&](int e) { put(0, __builtin_amdgcn_readfirstlane(e)); });
          fit_range<8, 16, 64>(r, [&](int e) { put(1, __builtin_amdgcn_readfirstlane(e)); });
          job_cvt<0, 8>(h, r, true);
#pragma unroll
          for (int f = 0; f < 8; ++f) *(lds_u4_p)(img + f * 1024 + ((f & 1) ? sl1 : sl0)) = h[f];
        } else {
          typedef _Float16 h4 __attribute__((ext_vector_type(4)));
          const int g = (lane >> 1) & 3, sw = ((lane >> 1) & 7);
          const lds_p a = img + (lane >> 3) * 1024 + g * 256 + (lane & 1) * 8;
          fit_range<0, 16, 32>(r, [&](int e) {
            put(0, __builtin_amdgcn_readlane(e, 0));
            put(1, __builtin_amdgcn_readlane(e, 32));
          });
          u2 w2[16];
          unsigned lo = ~0u;
#pragma unroll
          for (int k = 0; k < 16; ++k) {
            w2[k] = __builtin_bit_cast(u2, h4{(_Float16)r[k][0], (_Float16)r[k][1], (_Float16)r[k][2], (_Float16)r[k][3]});
            lo = min(lo, min(w2[k][0], w2[k][1]));
          }
          if (__builtin_expect(__ballot(lo == 0u) != 0ull, 0)) {
#pragma unroll
            for (int k = 0; k < 16; ++k) {
              w2[k][0] = w2[k][0] ? w2[k][0] : 0x80008000u;
              w2[k][1] = w2[k][1] ? w2[k][1] : 0x80008000u;
            }
          }
#pragma unroll
          for (int k = 0; k < 16; ++k) *(__attribute__((address_space(3))) u2*)(a + (k ^ sw) * 16) = w2[k];
        }
        if (cross && q == 31) {
          if (!call) call = next_call(ctr, lane);
          const unsigned* m = (const unsigned*)(myws + MK_OFF + (link - 3) * FL_STR);
          while (__hip_atomic_load(m, __ATOMIC_RELAXED, __HIP_MEMORY_SCOPE_AGENT) - (call - 1u) * 64u < 64u) {
            if (__builtin_amdgcn_s_memrealtime() - tstart > DEADLINE) {
              if (lane == 0) __hip_atomic_store(tmo, call, __ATOMIC_RELAXED, __HIP_MEMORY_SCOPE_AGENT);
              const u4 nan = {0x7E007E00u, 0x7E007E00u, 0x7E007E00u, 0x7E007E00u};
#pragma unroll
              for (int f = 0; f < 8; ++f) *(lds_u4_p)(img + f * 1024 + sl0) = nan;
              break;
            }
            __builtin_amdgcn_s_sleep(2);
          }
        }
        __builtin_amdgcn_fence(__ATOMIC_RELEASE, "workgroup");
        if (lane == 0) __hip_atomic_store(full + j, (unsigned)q + 1u, __ATOMIC_RELAXED, __HIP_MEMORY_SCOPE_WORKGROUP);
      }
      if (!call) next_call(ctr, lane);
    }
    return;
  }
  if (bi < 16) {
    const int sid = b;
    if (sid >= 64) {
      const int hid = sid - 64;
      const unsigned call = next_call(ctr, lane);
      while (__builtin_amdgcn_s_memrealtime() - tstart < TLOCAL) __builtin_amdgcn_s_sleep(2);
      if (wave < 4) {
        const int q = hid + 64 * (wave & 1);
        if (wave < 2 || q < NJ_P) {
          const Job jb = wave < 2 ? w_job(p, myws, q >> 5, q >> 5, q & 31, call) : x_job(p, myws, q >> 3, q & 7, call);
          f4 r[16];
          u4 h[8];
          CLEAR_VM();
          job_load(r, jb, lane);
          wait_vm<0>();
          pin16(r);
          fit_range<0, 8, 64>(r, [&](int e) { put_exp(jb.exps, __builtin_amdgcn_readfirstlane(e)); });
          fit_range<8, 16, 64>(r, [&](int e) { put_exp(jb.exps + 4, __builtin_amdgcn_readfirstlane(e)); });
          job_cvt<0, 8>(h, r, jb.isw);
          job_store(h, jb.dst, lane);
          if (jb.isw) {
            wait_vm<0>();
            st_flag(jb.flag, call);
          }
        }
        while (__builtin_amdgcn_s_memrealtime() - tstart < TCLEAR) __builtin_amdgcn_s_sleep(8);
        const int su = (p.tab[hid & 15] >> (8 * (hid >> 4))) & 127;
        char* a = myws + X_OFF + ((call + 2u) % 3u) * XB_B + su * XU_B + wave * (XU_B / 4) + lane * 16;
        const u4 z = {0u, 0u, 0u, 0u};
#pragma unroll 1
        for (int j = 0; j < 20; ++j) {
          if (j == 16) a = myws + W_OFF + ((call + 2u) % 3u) * WB_B + (4 + wave) * WS_B + hid * 4096 - 16384 + lane * 16;
          st16_sys<0>(a + j * 1024, z);
          __builtin_amdgcn_s_sleep(ZSTEP);
        }
        if (wave == 0 && hid < 10) st16_sys<0>(myws + E_OFF + ((call + 2u) % 3u) * EB_B + hid * 1024 + lane * 16, z);
      } else {
        const int col = ((idx + wave + rank) & 3) * 64;
        const int lo = ((lane >> 4) * SH + col + (lane & 15) * 4) * 4;
        const int o0 = (int)((p.otab[hid >> 2] >> (8 * (hid & 3))) & 127) * (16 * SH * 4);
        char* const out = (char*)p.out[rank];
        const u4 mk = {MARK, MARK, MARK, MARK};
#pragma unroll
        for (int j = 0; j < 4; ++j) st16_sys<0>(out + o0 + lo + j * 4 * SH * 4, mk);
        wait_vm<0>();
        if (lane == 0)
          __hip_atomic_fetch_add((unsigned*)(myws + MK_OFF + (wave - 4) * FL_STR), 1u, __ATOMIC_RELAXED,
                                 __HIP_MEMORY_SCOPE_AGENT);
        while (__builtin_amdgcn_s_memrealtime() - tstart < OPOLL0) __builtin_amdgcn_s_sleep(8);
        const __amdgpu_buffer_rsrc_t ro = __builtin_amdgcn_make_buffer_rsrc(out, 0, MM * SH * 4, 0x00020000);
        const unsigned* const one = lane == 0 ? (const unsigned*)(out + o0 + (15 * SH + col + 63) * 4) : tmo;
        auto give_up = [&](unsigned v1, u64 el) {
          if (__ballot(lane != 0 && v1 == call) == 0ull && el <= DEADLINE) return false;
          if (lane == 0) {
            __hip_atomic_store(tmo, call, __ATOMIC_RELAXED, __HIP_MEMORY_SCOPE_AGENT);
            p.out[rank][(size_t)rank * SH * SH] = __builtin_nanf("");
          }
          return true;
        };
        bool dead = false;
        for (;;) {
          const unsigned v1 = __hip_atomic_load(one, __ATOMIC_RELAXED, __HIP_MEMORY_SCOPE_SYSTEM);
          const u64 el = __builtin_amdgcn_s_memrealtime() - tstart;
          if ((dead = give_up(v1, el))) break;
          if (el > OFULL || __builtin_amdgcn_readfirstlane(v1) != MARK) break;
          __builtin_amdgcn_s_sleep(4);
        }
        auto fetch = [&](u4 (&v)[4]) {
          asm volatile("" ::: "memory");
#pragma unroll
          for (int j = 0; j < 4; ++j) v[j] = __builtin_amdgcn_raw_buffer_load_b128(ro, lo + j * 4 * SH * 4, o0, 17);
        };
        auto complete = [&](const u4 (&v)[4]) {
          bool left = false;
#pragma unroll
          for (int j = 0; j < 4; ++j)
#pragma unroll
            for (int k = 0; k < 4; ++k) left |= v[j][k] == MARK;
          return __ballot(left) == 0ull;
        };
        if (!dead) {
          u4 va[4];
          fetch(va);
#pragma unroll 1
          for (int k = 1; !complete(va); ++k) {
            if ((k & 7) == 0 && give_up(__hip_atomic_load(one, __ATOMIC_RELAXED, __HIP_MEMORY_SCOPE_SYSTEM),
                                        __builtin_amdgcn_s_memrealtime() - tstart)) break;
            fetch(va);
          }
        }
      }
    }
    return;
  }

  const unsigned call = next_call(ctr, lane);
  const int ci = bi - 16;
  const int g = (ci & 8) + xcd, slot = ci & 7;
  const int odst = slot < 4 ? rank : (grp ^ 4) + (slot - 4);
  float* const obase = p.out[odst & 7] + (slot < 4 ? slot : (idx + odst) & 3) * 64;
  const unsigned units = p.tab[g];
  const __amdgpu_buffer_rsrc_t rs = __builtin_amdgcn_make_buffer_rsrc(myws, 0, (int)FW_OFF, 0x00020000);
  const int voff = lane * 16;

  const lds_u32_p cnt = (lds_u32_p)(lds + 4 * 8192);
  if (threadIdx.x < 80) cnt[threadIdx.x] = 0u;
  __syncthreads();

  const bool remote = slot >= 4;
  h8 B[8][4];
  int ew = 0;
  const int eb = E_OFF + (call % 3u) * EB_B;
  bool ok = true;
  {
    const int sb = W_OFF + (call % 3u) * WB_B + slot * WS_B + wave * 32768;
    const unsigned* a = (const unsigned*)(myws + (lane >= 4 ? TM_OFF : remote ? sb + lane * JOB_B + PROBE
                                                  : FW_OFF + (slot * 32 + wave * 4 + lane) * FL_STR));
    const unsigned least = remote ? 1u : call;
#pragma unroll 1
    for (;;) {
      const unsigned v = __hip_atomic_load(a, __ATOMIC_RELAXED, __HIP_MEMORY_SCOPE_SYSTEM);
      if (__ballot(lane >= 4 && v == call) != 0ull) ok = false;
      if (__builtin_amdgcn_s_memrealtime() - tstart > DEADLINE) {
        if (lane == 0) __hip_atomic_store(tmo, call, __ATOMIC_RELAXED, __HIP_MEMORY_SCOPE_AGENT);
        ok = false;
      }
      if (ok && __ballot(lane < 4 && v < least) != 0ull) {
        __builtin_amdgcn_s_sleep(1);
        continue;
      }
      asm volatile("" ::: "memory");
      unsigned lo = ~0u;
      ew = __builtin_amdgcn_raw_buffer_load_b32(rs, (lane & 7) * 4, eb + 8192 + (slot * 32 + wave * 4) * 8, 17);
#pragma unroll
      for (int s = 0; s < 8; ++s)
#pragma unroll
        for (int e = 0; e < 4; ++e) {
          const u4 v4 = __builtin_amdgcn_raw_buffer_load_b128(rs, voff + e * 1024, sb + s * 4096, 17);
          B[s][e] = __builtin_bit_cast(h8, v4);
        }
      __builtin_amdgcn_sched_barrier(0);
#pragma unroll
      for (int s = 0; s < 8; ++s)
#pragma unroll
        for (int e = 0; e < 4; ++e) {
          const u4 v4 = __builtin_bit_cast(u4, B[s][e]);
          lo = min(min(lo, min(v4[0], v4[1])), min(v4[2], v4[3]));
        }
      if (!ok || __ballot(lo == 0u) == 0ull) break;
    }
    ok = __builtin_amdgcn_readfirstlane(ok) != 0;
  }

  const unsigned xb = X_OFF + (call % 3u) * XB_B + wave * JOB_B;
  const char* const probe = myws + (lane < 4 ? xb + ((units >> (8 * (lane & 3))) & 127) * XU_B + PROBE : TM_OFF);
  const unsigned release = remote ? REL0 + RELSTEP * (15 - g) : 0u;
  const unsigned vo = (voff & 0x300) * 4 + (voff & 0xF0);
  const unsigned pw = (unsigned)(size_t)(cnt + 16 + 8 * wave);
  const lds_u32_p look = cnt + (lane < 5 ? 16 + 8 * wave + lane : lane < 8 ? lane - 1 : 4);
  unsigned todo = 15u, rdy = 0u;
  u64 now = tstart;
#pragma unroll 1
  for (;;) {
    int red = -1;
    if (!rdy) {
      __builtin_amdgcn_s_setprio(0);
      if (todo) {
        unsigned keep;
        asm volatile("s_mov_b32 %0, m0\n\ts_mov_b32 m0, %2\n\ts_mov_b64 exec, %3\n\t"
                     "global_load_lds_dword %1, off sc0 sc1\n\ts_mov_b64 exec, -1\n\ts_mov_b32 m0, %0"
                     : "=&s"(keep) : "v"(probe), "s"(pw), "s"((u64)(todo | 16u)) : "memory");
        __builtin_amdgcn_s_sleep(2);
      }
      now = __builtin_amdgcn_s_memrealtime();
      const unsigned lv = __hip_atomic_load(look, __ATOMIC_RELAXED, __HIP_MEMORY_SCOPE_WORKGROUP);
      const unsigned sent = __builtin_amdgcn_readlane(lv, 5), held = __builtin_amdgcn_readlane(lv, 6);
      const unsigned tosum = __builtin_amdgcn_readlane(lv, 7);
      rdy = (unsigned)__ballot(lv != 0u) & todo;
      if (__builtin_amdgcn_readlane(lv, 4) == call) { ok = false; rdy = todo; }
      if (now - tstart > DEADLINE) {
        if (lane == 0) __hip_atomic_store(tmo, call, __ATOMIC_RELAXED, __HIP_MEMORY_SCOPE_AGENT);
        if (!todo && !tosum && !held) break;
        ok = false;
        rdy = todo;
      }
      if (!rdy) {
        if (tosum) {
          const unsigned bit = tosum & (0u - tosum);
          unsigned old = 0u;
          if (lane == 0) old = __hip_atomic_fetch_and(cnt + 6, ~bit, __ATOMIC_ACQUIRE, __HIP_MEMORY_SCOPE_WORKGROUP);
          if (!(__builtin_amdgcn_readfirstlane(old) & bit)) continue;
          red = __builtin_ctz(bit);
        } else if (held && (unsigned)(now - tstart) >= release + TSTEP * sent) {
          const unsigned bit = held & (0u - held);
          unsigned got = 0u;
          if (lane == 0) {
            unsigned exp = sent;
            if (__hip_atomic_compare_exchange_strong(cnt + 4, &exp, sent + 1u, __ATOMIC_RELAXED, __ATOMIC_RELAXED,
                                                     __HIP_MEMORY_SCOPE_WORKGROUP)) {
              got = __hip_atomic_fetch_and(cnt + 5, ~bit, __ATOMIC_ACQUIRE, __HIP_MEMORY_SCOPE_WORKGROUP) & bit;
              if (!got) __hip_atomic_fetch_sub(cnt + 4, 1u, __ATOMIC_RELAXED, __HIP_MEMORY_SCOPE_WORKGROUP);
            }
          }
          if (__builtin_amdgcn_readfirstlane(got)) {
            __builtin_amdgcn_fence(__ATOMIC_ACQUIRE, "workgroup");
            const int t = __builtin_ctz(bit);
            const float* buf = lds + t * 8192 + lane * 4;
            const char* o = sgpr_ptr((char*)obase + ((units >> (8 * t)) & 127) * (16 * SH * 4));
            f4 sum[4];
#pragma unroll
            for (int j = 0; j < 4; ++j) sum[j] = *(const f4*)(buf + j * 256);
#pragma unroll
            for (int j = 0; j < 4; ++j) st16_sys_s(o, vo + j * 4 * SH * 4, __builtin_bit_cast(u4, sum[j]));
          }
          continue;
        } else {
          if (!todo && sent == 4u) break;
          if (!todo) __builtin_amdgcn_s_sleep(4);
          continue;
        }
      }
    }
    if (rdy) {
      const int t = __builtin_ctz(rdy);
      __builtin_amdgcn_s_setprio(3);
      const int sb = __builtin_amdgcn_readfirstlane(xb + ((units >> (8 * t)) & 127) * XU_B);
      asm volatile("" ::: "memory");
      u4 A[8];
      const int ex = __builtin_amdgcn_raw_buffer_load_b32(rs, (lane & 1) * 4,
          __builtin_amdgcn_readfirstlane(eb + (((units >> (8 * t)) & 127) * 8 + wave) * 8), 17);
#pragma unroll
      for (int s = 0; s < 8; ++s)
        A[s] = __builtin_amdgcn_raw_buffer_load_b128(rs, voff + (s & 3) * 1024, sb + (s >> 2) * 4096, 17);
      __builtin_amdgcn_sched_barrier(0);
      f4 acc[4];
#pragma unroll
      for (int e = 0; e < 4; ++e) acc[e] = f4{0.f, 0.f, 0.f, 0.f};
      unsigned lo = ~0u;
#pragma unroll
      for (int s = 0; s < 8; ++s) {
#pragma unroll
        for (int e = 0; e < 4; ++e)
          acc[e] = __builtin_amdgcn_mfma_f32_16x16x32_f16(__builtin_bit_cast(h8, A[s]), B[s][e], acc[e], 0, 0, 0);
        lo = min(min(lo, min(A[s][0], A[s][1])), min(A[s][2], A[s][3]));
      }
      const bool missing = __ballot(lo == 0u) != 0ull;
      if (ok & missing) {
        if (__builtin_amdgcn_s_memrealtime() - tstart <= DEADLINE) continue;
        if (lane == 0) __hip_atomic_store(tmo, call, __ATOMIC_RELAXED, __HIP_MEMORY_SCOPE_AGENT);
        ok = false;
      }
      rdy &= rdy - 1u;
      todo &= ~(1u << t);
      if (__builtin_expect(__ballot((ex | ew) != 0) != 0ull, 0)) {
#pragma unroll
        for (int e = 0; e < 4; ++e) acc[e] = f4{0.f, 0.f, 0.f, 0.f};
#pragma unroll
        for (int s = 0; s < 8; ++s) {
          const int sc = __builtin_amdgcn_readlane(ex, s >> 2) + __builtin_amdgcn_readlane(ew, s);
#pragma unroll
          for (int e = 0; e < 4; ++e) {
            const f4 d = __builtin_amdgcn_mfma_f32_16x16x32_f16(__builtin_bit_cast(h8, A[s]), B[s][e], f4{0.f, 0.f, 0.f, 0.f}, 0, 0, 0);
#pragma unroll
            for (int r = 0; r < 4; ++r) acc[e][r] += __builtin_ldexpf(d[r], sc);
          }
          asm volatile("" : "+v"(acc[0]), "+v"(acc[1]), "+v"(acc[2]), "+v"(acc[3]));
          __builtin_amdgcn_sched_barrier(0);
        }
      }
      if (!ok) {
        const float nan = __builtin_nanf("");
#pragma unroll
        for (int e = 0; e < 4; ++e) acc[e] = f4{nan, nan, nan, nan};
      }
      float* slab = lds + t * 8192 + wave * 1024 + (lane >> 4) * 4 * 64 + (lane & 15) * 4;
#pragma unroll
      for (int r = 0; r < 4; ++r) *(f4*)(slab + r * 64) = f4{acc[0][r], acc[1][r], acc[2][r], acc[3][r]};
      unsigned old = 0;
      if (lane == 0) old = __hip_atomic_fetch_add(cnt + t, 1u, __ATOMIC_ACQ_REL, __HIP_MEMORY_SCOPE_WORKGROUP);
      if (__builtin_amdgcn_readfirstlane(old) == 7u) {
        if (todo) {
          if (lane == 0) __hip_atomic_fetch_or(cnt + 6, 1u << t, __ATOMIC_RELEASE, __HIP_MEMORY_SCOPE_WORKGROUP);
        } else {
          red = t;
        }
      }
    }
    if (red >= 0) {
      __builtin_amdgcn_fence(__ATOMIC_ACQUIRE, "workgroup");
      now = __builtin_amdgcn_s_memrealtime();
      const unsigned nsent = __builtin_amdgcn_readfirstlane(__hip_atomic_load(cnt + 4, __ATOMIC_RELAXED, __HIP_MEMORY_SCOPE_WORKGROUP));
      float* const buf = lds + red * 8192;
      f4 sum[4], pa[4][4];
#pragma unroll
      for (int hf = 0; hf < 2; ++hf) {
#pragma unroll
        for (int v = 0; v < 4; ++v)
#pragma unroll
          for (int j = 0; j < 4; ++j) pa[v][j] = *(const f4*)(buf + (hf * 4 + v) * 1024 + j * 256 + lane * 4);
        __builtin_amdgcn_sched_barrier(0);
#pragma unroll
        for (int j = 0; j < 4; ++j) {
          const f4 q = (pa[0][j] + pa[1][j]) + (pa[2][j] + pa[3][j]);
          sum[j] = hf ? sum[j] + q : q;
        }
        asm volatile("" : "+v"(sum[0]), "+v"(sum[1]), "+v"(sum[2]), "+v"(sum[3]));
      }
#pragma unroll
      for (int j = 0; j < 4; ++j) sum[j] = __builtin_elementwise_maximum(sum[j], f4{0.f, 0.f, 0.f, 0.f});
      asm volatile("" : "+v"(sum[0]), "+v"(sum[1]), "+v"(sum[2]), "+v"(sum[3]));
      if ((unsigned)(now - tstart) < release + (remote ? TSTEP * nsent : 0u)) {
#pragma unroll
        for (int j = 0; j < 4; ++j) *(f4*)(buf + j * 256 + lane * 4) = sum[j];
        __builtin_amdgcn_fence(__ATOMIC_RELEASE, "workgroup");
        if (lane == 0) __hip_atomic_fetch_or(cnt + 5, 1u << red, __ATOMIC_RELAXED, __HIP_MEMORY_SCOPE_WORKGROUP);
      } else {
        const char* o = sgpr_ptr((char*)obase + ((units >> (8 * red)) & 127) * (16 * SH * 4));
#pragma unroll
        for (int j = 0; j < 4; ++j) st16_sys_s(o, vo + j * 4 * SH * 4, __builtin_bit_cast(u4, sum[j]));
        if (lane == 0) __hip_atomic_fetch_add(cnt + 4, 1u, __ATOMIC_RELAXED, __HIP_MEMORY_SCOPE_WORKGROUP);
      }
    }
  }
}

}

extern "C" void kernel_launch(int rank, int nranks, ncclComm_t comm, hipStream_t stream,
                              void* const* d_in, const int* in_sizes, int n_in,
                              void* d_out, int out_size,
                              const int* in_dtypes, int out_dtype,
                              void* const* d_out_all,
                              void* const* d_ws_all, size_t ws_bytes) {
  (void)comm; (void)in_dtypes; (void)out_dtype; (void)d_out;
  if (nranks != NR || (unsigned)rank >= (unsigned)NR || n_in != 2 || in_sizes[0] != SH * KK ||
      in_sizes[1] != KK * SH || out_size != MM * SH || ws_bytes < WS_END) {
    fprintf(stderr, "unexpected problem shape\n");
    return;
  }
  Params p;
  p.x = (const float*)d_in[0];
  p.w = (const float*)d_in[1];
  for (int i = 0; i < NR; ++i) { p.ws[i] = (char*)d_ws_all[i]; p.out[i] = (float*)d_out_all[i]; }
  p.myws = p.ws[rank];
  p.rank = rank;
  p.pad = 0;
  int ent[64], key[64], n = 0, m = 0;
  for (int g = 0; g < 16; ++g) p.otab[g] = 0;
  for (int s = 0; s < NR; ++s) {
    const bool same = ((s ^ rank) & 4) == 0;
    for (int u = 0; u < 16; ++u) {
      if (same != (u < NP)) { p.otab[m >> 2] |= (unsigned)(s * 16 + u) << (8 * (m & 3)); ++m; continue; }
      const int k = s == rank ? 0 : (same ? 10 * (u + 1) : 50 + 10 * (u - NP));
      int j = n++;
      for (; j > 0 && key[j - 1] > k; --j) { key[j] = key[j - 1]; ent[j] = ent[j - 1]; }
      key[j] = k; ent[j] = s * 16 + u;
    }
  }
  for (int g = 0; g < 16; ++g)
    p.tab[g] = (unsigned)ent[g] | (unsigned)ent[16 + g] << 8 | (unsigned)ent[32 + g] << 16 | (unsigned)ent[48 + g] << 24;
  hipLaunchKernelGGL(ag_gemm_kernel, dim3(NBLOCKS), dim3(NTHREADS), 0, stream, p);
  HIP_CHECK(hipGetLastError());
}
```
